```python
import numpy as np

kernel_name = "gpu_gemm_ar_m4096_k4096_n4096_e4m3e4m3_dqsilu"


ABSMAX_RTOL = 0.03

COMPUTE_DTYPE = "float32"
EPILOGUE = "dequant_silu"
IN_QUANT = None
OUT_QUANT = None
IN_FP8 = "e4m3e4m3"
IN_DTYPES = {"x": "e4m3", "w_mat": "e4m3",
             "scale_x": "float32", "scale_w": "float32"}
WORLD_SIZE = 8

def _snap_fp8(x, mant_bits, max_abs, min_normal):
    a = np.abs(x.astype(np.float64))
    _m, _e = np.frexp(a)
    q = np.where(a >= min_normal, np.ldexp(1.0, _e - 1 - mant_bits),
                 min_normal * 2.0 ** -mant_bits)
    snapped = np.minimum(np.round(a / q) * q, max_abs)
    return (np.sign(x) * snapped).astype(np.float32)

def setup_inputs():

    x = _snap_fp8(np.random.randn(4096, 4096).astype(np.float32), 3, 448.0, 2.0 ** -6)
    w_mat = _snap_fp8(np.random.randn(4096, 4096).astype(np.float32), 3, 448.0, 2.0 ** -6)
    scale_x = (0.02 * (1.0 + np.random.rand(1))).astype(np.float32)
    scale_w = (0.02 * (1.0 + np.random.rand(1))).astype(np.float32)
    return {"x": x, "w_mat": w_mat, "scale_x": scale_x, "scale_w": scale_w}

def reference(x, w_mat, scale_x, scale_w):
    acc = (x @ w_mat).astype(np.float32)
    y = (acc * (scale_x * scale_w)).astype(np.float32)
    return (y / (1.0 + np.exp(-np.clip(y, -60.0, 60.0)))).astype(np.float32)
```

```cpp
#include <hip/hip_runtime.h>
#include <rccl/rccl.h>
#include <cstdio>

#define HIP_CHECK(expr) do { hipError_t _e = (expr); if (_e != hipSuccess) { \
    fprintf(stderr, "HIP error %s at %s:%d\n", hipGetErrorString(_e), __FILE__, __LINE__); \
    return; } } while (0)

namespace {

typedef unsigned u32x4 __attribute__((ext_vector_type(4)));
typedef int i32x8 __attribute__((ext_vector_type(8)));
typedef float f32x4 __attribute__((ext_vector_type(4)));
#define LAS __attribute__((address_space(3)))

constexpr int NR = 8;
constexpr int MDIM = 4096, NDIM = 4096, KSH = 512;
constexpr int NPULL = 28;
constexpr int GEMM0 = 32;
constexpr int NREP = 8;
constexpr int NWG = 256;
constexpr int NTM = 14;
constexpr int A_BYTES = 19 * 2048, B_BYTES = 16 * 2048, BUF_BYTES = A_BYTES + B_BYTES;
constexpr int CTL_OFF = 2 * BUF_BYTES;
constexpr int LDS_BYTES = CTL_OFF + 256;
constexpr size_t OP_BYTES = 256 * 2048;
constexpr size_t KT_BYTES = 2 * OP_BYTES;
constexpr size_t SHARD_BYTES = 4 * KT_BYTES;
constexpr size_t WS_SRC = 0;
constexpr size_t WS_DST = 2 * SHARD_BYTES;
constexpr size_t WS_READY = WS_DST + NR * SHARD_BYTES;
constexpr size_t WS_PROG = WS_READY + NR * 256 * 4;
constexpr size_t WS_TICKET = WS_PROG + 4096;
constexpr size_t WS_TMO = WS_TICKET + 16 * 128;
constexpr size_t WS_STAMP = WS_TMO + 1024;
constexpr size_t WS_END = WS_STAMP + 256 * 64 * 8;
constexpr unsigned long long DEADLINE = 20000000ull;

struct Params {
  const unsigned char* x; const unsigned char* w; const float* sx; const float* sw; float* out;
  char* const* ws_all; int rank; int pad;
};

#define SB() __builtin_amdgcn_sched_barrier(0)
__device__ __forceinline__ unsigned long long now() { return __builtin_amdgcn_s_memrealtime(); }
__device__ __forceinline__ void st16_sys(void* p, u32x4 v) { asm volatile("global_store_dwordx4 %0, %1, off sc0 sc1\n\ts_nop 1" :: "v"(p), "v"(v) : "memory"); }
__device__ __forceinline__ void st4_sys(void* p, unsigned v) { asm volatile("global_store_dword %0, %1, off sc0 sc1\n\ts_nop 1" :: "v"(p), "v"(v) : "memory"); }
__device__ __forceinline__ unsigned ld4_sys(const void* p) { unsigned v; asm volatile("global_load_dword %0, %1, off sc0 sc1\n\ts_waitcnt vmcnt(0)" : "=&v"(v) : "v"(p) : "memory"); return v; }
__device__ __forceinline__ int tile_start(int pm) { return (pm * 256) / NTM; }

__device__ __forceinline__ bool give_up(unsigned long long t0, unsigned* tmo, unsigned call) {
  if ((unsigned)__builtin_amdgcn_readfirstlane(ld4_sys(tmo)) == call) return true;
  if (now() - t0 > DEADLINE) { st4_sys(tmo, call); return true; }
  return false;
}

#define STAMP(i) do {} while (0)
#define STAMPW(i) do {} while (0)

#define R_B0 "v[200:207]"
#define R_B1 "v[208:215]"
#define R_B2 "v[216:223]"
#define R_B3 "v[224:231]"
#define R_AX "v[232:239]"
#define R_AY "v[240:247]"
#define RD_(f, LO, HI, ALL, addr, OFF) asm volatile("ds_read_b128 " LO ", %1 offset:%c2\n\tds_read_b128 " HI ", %1 offset:%c3" : "={" ALL "}"(f) : "v"(addr), "i"(OFF), "i"((OFF) + 1024))
#define RD_B0(addr, OFF) RD_(r.b0, "v[200:203]", "v[204:207]", R_B0, addr, OFF)
#define RD_B1(addr, OFF) RD_(r.b1, "v[208:211]", "v[212:215]", R_B1, addr, OFF)
#define RD_B2(addr, OFF) RD_(r.b2, "v[216:219]", "v[220:223]", R_B2, addr, OFF)
#define RD_B3(addr, OFF) RD_(r.b3, "v[224:227]", "v[228:231]", R_B3, addr, OFF)
#define RD_AX(addr, OFF) RD_(r.ax, "v[232:235]", "v[236:239]", R_AX, addr, OFF)
#define RD_AY(addr, OFF) RD_(r.ay, "v[240:243]", "v[244:247]", R_AY, addr, OFF)
#define WL(N) asm volatile("s_waitcnt lgkmcnt(" #N ")")
#define MM(ACC, RB, vb, RA, va) asm volatile("v_mfma_f32_16x16x128_f8f6f4 %0, " RB ", " RA ", %0" : "+v"(ACC) : "{" RB "}"(vb), "{" RA "}"(va))
#define MMA4X(m) do { MM(acc[m][0], R_B0, r.b0, R_AX, r.ax); MM(acc[m][1], R_B1, r.b1, R_AX, r.ax); MM(acc[m][2], R_B2, r.b2, R_AX, r.ax); MM(acc[m][3], R_B3, r.b3, R_AX, r.ax); } while (0)
#define MMA4Y(m) do { MM(acc[m][0], R_B0, r.b0, R_AY, r.ay); MM(acc[m][1], R_B1, r.b1, R_AY, r.ay); MM(acc[m][2], R_B2, r.b2, R_AY, r.ay); MM(acc[m][3], R_B3, r.b3, R_AY, r.ay); } while (0)
struct Regs { i32x8 b0, b1, b2, b3, ax, ay; };
__device__ __forceinline__ void prime_reads(Regs& r, unsigned pa, unsigned pb) {
  RD_AX(pa, 0); RD_B0(pb, 0); RD_B1(pb, 2048); RD_B2(pb, 4096); RD_B3(pb, 6144); RD_AY(pa, 2048);
}
__device__ __forceinline__ void dma(const char* sbase, unsigned voff, unsigned ldsa) {
  asm volatile("s_mov_b32 m0, %2\n\ts_nop 0\n\tglobal_load_lds_dwordx4 %0, %1 sc0 sc1" :: "v"(voff), "s"(sbase), "s"(ldsa) : "memory");
}
__device__ __forceinline__ void stage_all(const char* ga, const char* gb, unsigned voff, unsigned nl, bool a4) {
  const unsigned la = nl, lb = nl + A_BYTES;
  dma(gb, voff, lb); dma(gb + 8192, voff, lb + 8192); dma(gb + 2 * 8192, voff, lb + 2 * 8192); dma(gb + 3 * 8192, voff, lb + 3 * 8192);
  dma(ga, voff, la); dma(ga + 8192, voff, la + 8192); dma(ga + 2 * 8192, voff, la + 2 * 8192); dma(ga + 3 * 8192, voff, la + 3 * 8192);
  if (a4) dma(ga + 4 * 8192, voff, la + 4 * 8192);
}
__device__ __forceinline__ float silu(float acc, float s) {
  const float y = acc * s;
  const float c = __builtin_amdgcn_fmed3f(y, -60.0f, 60.0f);
  const float e = __builtin_amdgcn_exp2f(c * -1.44269504088896340736f);
  return y * __builtin_amdgcn_rcpf(1.0f + e);
}

__device__ __forceinline__ unsigned chunk_off(int q, int pi) { const int p = q >> 5, op = q & 1, c = (((q >> 1) & 15) + 2 * pi) & 15; return (unsigned)(p * KT_BYTES + op * OP_BYTES + c * 32768); }
__device__ __forceinline__ int chunk_pos(int p, int op, int c, int pi) { return p * 32 + 2 * ((c - 2 * pi) & 15) + op; }

__global__ __launch_bounds__(512, 2) void fused_kernel(Params P) {
  extern __shared__ __attribute__((aligned(16))) unsigned char smem[];
  LAS unsigned char* lds = (LAS unsigned char*)smem;
  LAS unsigned* ctl = (LAS unsigned*)(lds + CTL_OFF);
  const int tid = threadIdx.x, lane = tid & 63, wid = __builtin_amdgcn_readfirstlane(tid >> 6);
  const int b = blockIdx.x, rank = P.rank;
  char* myws = P.ws_all[rank];
  unsigned* tmo = (unsigned*)(myws + WS_TMO);
  STAMP(0);

  unsigned ticket = 0;
  if (tid == 448) ticket = __hip_atomic_fetch_add((unsigned*)(myws + WS_TICKET + (b & 15) * 128), 1u, __ATOMIC_RELAXED, __HIP_MEMORY_SCOPE_AGENT);
  {
    const int p = b >> 6, G = b & 63;
    u32x4 o[4];
    if (wid < 2) {
      const int c = lane & 15, q = lane >> 4;
      const unsigned char* src = P.w + (size_t)(p * 128 + wid * 64 + q * 16) * NDIM + G * 64 + c * 4;
      unsigned d[16];
#pragma unroll
      for (int j = 0; j < 16; ++j) d[j] = *(const unsigned*)(src + (size_t)j * NDIM);
#pragma unroll
      for (int jj = 0; jj < 4; ++jj) {
        const unsigned a0 = d[4 * jj], a1 = d[4 * jj + 1], a2 = d[4 * jj + 2], a3 = d[4 * jj + 3];
        const unsigned t0 = __builtin_amdgcn_perm(a1, a0, 0x05010400u), t1 = __builtin_amdgcn_perm(a1, a0, 0x07030602u);
        const unsigned t2 = __builtin_amdgcn_perm(a3, a2, 0x05010400u), t3 = __builtin_amdgcn_perm(a3, a2, 0x07030602u);
        o[0][jj] = __builtin_amdgcn_perm(t2, t0, 0x05040100u); o[1][jj] = __builtin_amdgcn_perm(t2, t0, 0x07060302u);
        o[2][jj] = __builtin_amdgcn_perm(t3, t1, 0x05040100u); o[3][jj] = __builtin_amdgcn_perm(t3, t1, 0x07060302u);
      }
    } else if (wid < 6) {
      const int fr = lane & 15, fq = lane >> 4;
      const unsigned char* src = P.x + (size_t)((4 * G + wid - 2) * 16 + fr) * KSH + p * 128 + fq * 16;
      o[0] = *(const u32x4*)src; o[1] = *(const u32x4*)(src + 64);
    }
    if (tid == 448) ctl[4] = ticket / 16 + 1;
    __syncthreads();
    const unsigned call_ = ctl[4];
    char* dst = myws + WS_SRC + (size_t)(call_ & 1) * SHARD_BYTES + (size_t)p * KT_BYTES;
    if (wid < 2) {
#pragma unroll
      for (int i = 0; i < 4; ++i) st16_sys(dst + OP_BYTES + (size_t)(4 * G + i) * 2048 + wid * 1024 + lane * 16, o[i]);
    } else if (wid < 6) {
      char* d2 = dst + (size_t)(4 * G + wid - 2) * 2048 + lane * 16;
      st16_sys(d2, o[0]); st16_sys(d2 + 1024, o[1]);
    }
    asm volatile("s_waitcnt vmcnt(0)\n\ts_barrier" ::: "memory");
    if (tid < NR) st4_sys(P.ws_all[tid] + WS_READY + (size_t)(rank * 256 + b) * 4, call_);
  }
  const unsigned call = (unsigned)__builtin_amdgcn_readfirstlane(ctl[4]);
  STAMP(1);

  if (b < GEMM0) {
    if (b >= NPULL) return;
    const int pi = b % 7, w4 = b / 7, peer = (rank + 1 + pi) & 7;
    if (tid < 32) ctl[8 + tid] = 0;
    if (wid == 0) {
      const unsigned* fl = (const unsigned*)(myws + WS_READY) + peer * 256 + lane * 4;
      const unsigned long long t0 = now();
      for (;;) {
        u32x4 v; asm volatile("global_load_dwordx4 %0, %1, off sc0 sc1\n\ts_waitcnt vmcnt(0)" : "=&v"(v) : "v"(fl) : "memory");
        if (__all(v[0] >= call && v[1] >= call && v[2] >= call && v[3] >= call)) break;
        if (give_up(t0, tmo, call)) break;
        __builtin_amdgcn_s_sleep(4);
      }
    }
    asm volatile("s_waitcnt vmcnt(0) lgkmcnt(0)\n\ts_barrier" ::: "memory");
    STAMP(2);
    const char* src = P.ws_all[peer] + WS_SRC + (size_t)(call & 1) * SHARD_BYTES + wid * 4096 + lane * 16;
    char* dst = myws + WS_DST + (size_t)peer * SHARD_BYTES + wid * 4096 + lane * 16;
    unsigned* prog = (unsigned*)(myws + WS_PROG + (lane & (NREP - 1)) * 128) + pi * 4 + w4;
    const unsigned pbase = call * 64;
    const unsigned arr = (unsigned)(size_t)lds + CTL_OFF + 32;
    auto soff = [&](int u) -> unsigned { return chunk_off(4 * (u >> 1) + w4, pi) + (u & 1) * 2048; };
    i32x8 va, vb;
    asm volatile("global_load_dwordx4 v[200:203], %1, off sc0 sc1\n\tglobal_load_dwordx4 v[204:207], %1, off offset:1024 sc0 sc1\n\ts_waitcnt vmcnt(0)"
                 : "={v[200:207]}"(va) : "v"(src + soff(0)) : "memory");
    auto arrive = [&](int t) {
      unsigned old;
      asm volatile("ds_add_rtn_u32 %0, %1, %2\n\ts_waitcnt lgkmcnt(0)" : "=&v"(old) : "v"(arr + t * 4), "v"(1u) : "memory");
      if (__builtin_amdgcn_readfirstlane(old) == 7) return true;
      return false;
    };
#define PULL_STEP(CUR, CLO, CHI, CALL, NXT, NLO, NHI, NALL, u_) do { \
      const int un_ = (u_) + 1 < 64 ? (u_) + 1 : (u_); \
      asm volatile("global_load_dwordx4 " NLO ", %1, off sc0 sc1\n\tglobal_load_dwordx4 " NHI ", %1, off offset:1024 sc0 sc1\n\t" \
                   "global_store_dwordx4 %2, " CLO ", off sc0 sc1\n\tglobal_store_dwordx4 %2, " CHI ", off offset:1024 sc0 sc1\n\t" \
                   "s_waitcnt vmcnt(2)" : "={" NALL "}"(NXT) : "v"(src + soff(un_)), "v"(dst + soff(u_)), "{" CALL "}"(CUR) : "memory"); \
    } while (0)
    auto publish = [&](int t) {
      bool last = false;
      if (lane == 0) last = arrive(t);
      last = __builtin_amdgcn_readfirstlane((int)last) != 0;
      if (last && lane < NREP) asm volatile("global_atomic_umax %0, %1, off sc1" :: "v"(prog), "v"(pbase + t + 1) : "memory");
    };
    for (int u = 0; u < 64; u += 2) {
      PULL_STEP(va, "v[200:203]", "v[204:207]", "v[200:207]", vb, "v[208:211]", "v[212:215]", "v[208:215]", u);
      if (u > 0) publish((u >> 1) - 1);
      PULL_STEP(vb, "v[208:211]", "v[212:215]", "v[208:215]", va, "v[200:203]", "v[204:207]", "v[200:207]", u + 1);
    }
#undef PULL_STEP
    asm volatile("s_waitcnt vmcnt(0)" ::: "memory");
    publish(31);
    STAMPW(10 + wid);
    return;
  }

  const int g = b - GEMM0, xcd = g & 7, idx = g >> 3;
  const int pm = (xcd & 1) * 7 + idx % 7, pn = (xcd >> 1) * 4 + idx / 7;
  const int g0 = tile_start(pm), g1 = tile_start(pm + 1), wr = wid >> 2, wc = wid & 3;
  const bool a4 = wid < (g1 - g0) * 2 - 32;
  const char* own = myws + WS_SRC + (size_t)(call & 1) * SHARD_BYTES;
  const char* dstb = myws + WS_DST;
  const size_t aoff = (size_t)g0 * 2048, boff = OP_BYTES + (size_t)pn * B_BYTES;
  auto kt_base = [&](int kt) -> const char* { const int s = kt >> 2, p = kt & 3; return (s == rank ? own : dstb + (size_t)s * SHARD_BYTES) + (size_t)p * KT_BYTES; };

  unsigned pollA, pollB, pollC, thrA, thrB, thrC;
  {
    const int kt = lane & 31, s = kt >> 2, p = kt & 3, pi0 = (s - rank - 1) & 7, pi = pi0 < 7 ? pi0 : 0;
    const int qa = chunk_pos(p, 0, g0 >> 4, pi), qb = chunk_pos(p, 0, (g1 - 1) >> 4, pi), qc = chunk_pos(p, 1, pn, pi);
    const unsigned base = (unsigned)WS_PROG + (unsigned)(g & (NREP - 1)) * 128 + pi * 16;
    pollA = base + (qa & 3) * 4; pollB = base + (qb & 3) * 4; pollC = base + (qc & 3) * 4;
    thrA = call * 64 + (qa >> 2) + 1; thrB = call * 64 + (qb >> 2) + 1; thrC = call * 64 + (qc >> 2) + 1;
  }
  const unsigned ownmask = 0xFu << (4 * rank);

  f32x4 acc[10][4];
#pragma unroll
  for (int m = 0; m < 10; ++m)
#pragma unroll
    for (int n = 0; n < 4; ++n) acc[m][n] = (f32x4){0.f, 0.f, 0.f, 0.f};

  if (wid == 0) {
    const unsigned* fl = (const unsigned*)(myws + WS_READY) + rank * 256 + lane * 4;
    const unsigned long long t0 = now();
    for (;;) {
      u32x4 v; asm volatile("global_load_dwordx4 %0, %1, off sc0 sc1\n\ts_waitcnt vmcnt(0)" : "=&v"(v) : "v"(fl) : "memory");
      if (__all(v[0] >= call && v[1] >= call && v[2] >= call && v[3] >= call)) break;
      if (give_up(t0, tmo, call)) break;
      __builtin_amdgcn_s_sleep(4);
    }
  }
  asm volatile("s_waitcnt vmcnt(0) lgkmcnt(0)\n\ts_barrier" ::: "memory");
  STAMP(2);

  const unsigned lbase = (unsigned)(size_t)lds;
  const unsigned voff = wid * 1024 + lane * 16;
  unsigned pa = lbase + wr * (10 * 2048) + lane * 16, pb = lbase + A_BYTES + wc * (4 * 2048) + lane * 16;
  unsigned npa = pa + BUF_BYTES, npb = pb + BUF_BYTES;
  const unsigned ctl_addr = lbase + CTL_OFF;
  unsigned ready = ownmask, taken = 0;
  Regs r;
  { const int k = __builtin_ctz(ready); taken |= 1u << k; const char* kb = kt_base(k);
    stage_all(kb + aoff, kb + boff, voff, lbase + wid * 1024, a4);
    asm volatile("s_waitcnt vmcnt(0)\n\ts_barrier" ::: "memory");
    prime_reads(r, pa, pb); }
  int cur = 0;
  for (int n = 0; n < 32; ++n) {
    const unsigned cand = ready & ~taken;
    const bool stage = cand != 0;
    const int k = stage ? __builtin_ctz(cand) : 0;
    if (stage) taken |= 1u << k;
    const char* kb = kt_base(k);
    const char* ga = kb + aoff; const char* gb = kb + boff;
    const unsigned nl = lbase + (cur ^ 1) * BUF_BYTES + wid * 1024;
    const unsigned la = nl, lb = nl + A_BYTES;
    unsigned q0, q1, q2;
    const unsigned ctl_cur = ctl_addr + (n & 1) * 4;
#define DB(i) dma(gb + (i) * 8192, voff, lb + (i) * 8192)
#define DA(i) dma(ga + (i) * 8192, voff, la + (i) * 8192)
    if (wid == 0) asm volatile("global_load_dword %0, %3, %6 sc1\n\tglobal_load_dword %1, %4, %6 sc1\n\tglobal_load_dword %2, %5, %6 sc1"
                               : "=&v"(q0), "=&v"(q1), "=&v"(q2) : "v"(pollA), "v"(pollB), "v"(pollC), "s"(myws) : "memory");
    if (stage) { DB(0); DB(1); }
    __builtin_amdgcn_s_setprio(1);
    WL(8); MM(acc[0][0], R_B0, r.b0, R_AX, r.ax);
    WL(6); MM(acc[0][1], R_B1, r.b1, R_AX, r.ax);
    WL(4); MM(acc[0][2], R_B2, r.b2, R_AX, r.ax);
    WL(2); MM(acc[0][3], R_B3, r.b3, R_AX, r.ax);
    RD_AX(pa, 2 * 2048);
    if (stage) { DB(2); DB(3); }
    WL(2); MMA4Y(1); RD_AY(pa, 3 * 2048);
    if (stage) { DA(0); DA(1); }
    WL(2); MMA4X(2); RD_AX(pa, 4 * 2048);
    if (stage) { DA(2); }
    WL(2); MMA4Y(3); RD_AY(pa, 5 * 2048);
    if (stage) { DA(3); }
    WL(2); MMA4X(4); RD_AX(pa, 6 * 2048);
    if (stage && a4) { DA(4); }
    WL(2); MMA4Y(5); RD_AY(pa, 7 * 2048);
    WL(2); MMA4X(6); RD_AX(pa, 8 * 2048);
    if (wid == 0) {
      if (stage) asm volatile("s_waitcnt vmcnt(9)" : "+v"(q0), "+v"(q1), "+v"(q2) :: "memory");
      else asm volatile("s_waitcnt vmcnt(0)" : "+v"(q0), "+v"(q1), "+v"(q2) :: "memory");
      const unsigned m = (unsigned)__ballot(q0 >= thrA && q1 >= thrB && q2 >= thrC) | ownmask;
      if (lane == 0) asm volatile("ds_write_b32 %0, %1" :: "v"(ctl_cur), "v"(m) : "memory");
    }
    WL(2); MMA4Y(7); RD_AY(pa, 9 * 2048);
    WL(2); MMA4X(8);
    asm volatile("s_waitcnt vmcnt(0) lgkmcnt(0)\n\ts_barrier" ::: "memory");
    unsigned mreg;
    asm volatile("ds_read_b32 %0, %1" : "=v"(mreg) : "v"(ctl_cur));
    RD_AX(npa, 0);
    asm volatile(
      "s_cmp_lg_u32 %[wr], 0\n\t"
      "s_cbranch_scc1 1f\n\t"
      "v_mfma_f32_16x16x128_f8f6f4 %[c0], v[200:207], v[240:247], %[c0]\n\t"
      "ds_read_b128 v[200:203], %[pb] offset:0\n\tds_read_b128 v[204:207], %[pb] offset:1024\n\t"
      "v_mfma_f32_16x16x128_f8f6f4 %[c1], v[208:215], v[240:247], %[c1]\n\t"
      "ds_read_b128 v[208:211], %[pb] offset:2048\n\tds_read_b128 v[212:215], %[pb] offset:3072\n\t"
      "v_mfma_f32_16x16x128_f8f6f4 %[c2], v[216:223], v[240:247], %[c2]\n\t"
      "ds_read_b128 v[216:219], %[pb] offset:4096\n\tds_read_b128 v[220:223], %[pb] offset:5120\n\t"
      "v_mfma_f32_16x16x128_f8f6f4 %[c3], v[224:231], v[240:247], %[c3]\n\t"
      "ds_read_b128 v[224:227], %[pb] offset:6144\n\tds_read_b128 v[228:231], %[pb] offset:7168\n\t"
      "s_branch 2f\n"
      "1:\n\t"
      "ds_read_b128 v[200:203], %[pb] offset:0\n\tds_read_b128 v[204:207], %[pb] offset:1024\n\t"
      "ds_read_b128 v[208:211], %[pb] offset:2048\n\tds_read_b128 v[212:215], %[pb] offset:3072\n\t"
      "ds_read_b128 v[216:219], %[pb] offset:4096\n\tds_read_b128 v[220:223], %[pb] offset:5120\n\t"
      "ds_read_b128 v[224:227], %[pb] offset:6144\n\tds_read_b128 v[228:231], %[pb] offset:7168\n"
      "2:"
      : [c0] "+v"(acc[9][0]), [c1] "+v"(acc[9][1]), [c2] "+v"(acc[9][2]), [c3] "+v"(acc[9][3]),
        "+{v[200:207]}"(r.b0), "+{v[208:215]}"(r.b1), "+{v[216:223]}"(r.b2), "+{v[224:231]}"(r.b3)
      : [wr] "s"(wr), [pb] "v"(npb), "{v[240:247]}"(r.ay) : "scc");
    RD_AY(npa, 2048);
    __builtin_amdgcn_s_setprio(0);
    asm volatile("s_waitcnt lgkmcnt(12)" : "+v"(mreg));
    ready = (unsigned)__builtin_amdgcn_readfirstlane(mreg);
#undef DA
#undef DB
    if (n == 31) break;
    if (!stage) {
      if ((ready & ~taken) == 0) {
        if (wid == 0) {
          const unsigned long long t0 = now();
          for (;;) {
            asm volatile("global_load_dword %0, %3, %6 sc1\n\tglobal_load_dword %1, %4, %6 sc1\n\tglobal_load_dword %2, %5, %6 sc1\n\ts_waitcnt vmcnt(0)"
                         : "=&v"(q0), "=&v"(q1), "=&v"(q2) : "v"(pollA), "v"(pollB), "v"(pollC), "s"(myws) : "memory");
            unsigned m = (unsigned)__ballot(q0 >= thrA && q1 >= thrB && q2 >= thrC) | ownmask;
            if ((m & ~taken) == 0 && give_up(t0, tmo, call)) m = ~0u;
            if ((m & ~taken) != 0) { if (lane == 0) asm volatile("ds_write_b32 %0, %1" :: "v"(ctl_addr + 8), "v"(m) : "memory"); break; }
            __builtin_amdgcn_s_sleep(4);
          }
        }
        asm volatile("s_waitcnt lgkmcnt(0)\n\ts_barrier\n\tds_read_b32 %0, %1\n\ts_waitcnt lgkmcnt(0)" : "=v"(mreg) : "v"(ctl_addr + 8) : "memory");
        ready = (unsigned)__builtin_amdgcn_readfirstlane(mreg);
      }
      const int k2 = __builtin_ctz(ready & ~taken); taken |= 1u << k2;
      const char* kb2 = kt_base(k2);
      stage_all(kb2 + aoff, kb2 + boff, voff, nl, a4);
      asm volatile("s_waitcnt vmcnt(0)\n\ts_barrier" ::: "memory");
      prime_reads(r, npa, npb);
    }
    { unsigned t = pa; pa = npa; npa = t; t = pb; pb = npb; npb = t; cur ^= 1; }
  }
  asm volatile("s_waitcnt lgkmcnt(0)\n\ts_nop 15\n\ts_nop 15\n\ts_barrier" ::: "memory");
  STAMP(3);

  {
    float s = P.sx[0] * P.sw[0];
    const bool bad = (unsigned)__builtin_amdgcn_readfirstlane(ld4_sys(tmo)) == call;
    if (bad) s = __builtin_nanf("");
    LAS unsigned char* reg = lds + wid * (2 * 16 * 272);
    const int fr = lane & 15, fq = lane >> 4;
#pragma unroll
    for (int m = 0; m < 10; ++m) {
      const int gg = g0 + wr * 10 + m;
      if (gg < g1 && (wr == 0 || m < 9)) {
        LAS unsigned char* rr = reg + (m & 1) * (16 * 272);
#pragma unroll
        for (int j = 0; j < 4; ++j) {
          f32x4 v = {silu(acc[m][0][j], s), silu(acc[m][1][j], s), silu(acc[m][2][j], s), silu(acc[m][3][j], s)};
          *(LAS f32x4*)(rr + fr * 272 + fq * 64 + j * 16) = v;
        }
        float* o = P.out + (size_t)(gg * 16 + fq) * NDIM + pn * 256 + wc * 64 + fr * 4;
#pragma unroll
        for (int r4 = 0; r4 < 4; ++r4) {
          f32x4 v = *(const LAS f32x4*)(rr + (r4 * 4 + fq) * 272 + fr * 16);
          *(f32x4*)(o + (size_t)r4 * 4 * NDIM) = v;
        }
      }
    }
  }
}

}

extern "C" void kernel_launch(int rank, int nranks, ncclComm_t comm, hipStream_t stream,
                              void* const* d_in, const int* in_sizes, int n_in,
                              void* d_out, int out_size, const int* in_dtypes, int out_dtype,
                              void* const* d_out_all, void* const* d_ws_all, size_t ws_bytes) {
  (void)comm; (void)in_sizes; (void)n_in; (void)out_size; (void)in_dtypes; (void)out_dtype; (void)d_out_all;
  if (nranks != NR || (unsigned)rank >= (unsigned)NR || ws_bytes < WS_END) { fprintf(stderr, "unsupported configuration\n"); return; }
  static bool attr_done[NR] = {};
  if (!attr_done[rank]) {
    HIP_CHECK(hipFuncSetAttribute((const void*)fused_kernel, hipFuncAttributeMaxDynamicSharedMemorySize, LDS_BYTES));
    attr_done[rank] = true;
  }
  Params p;
  p.x = (const unsigned char*)d_in[0]; p.w = (const unsigned char*)d_in[1]; p.sx = (const float*)d_in[2]; p.sw = (const float*)d_in[3];
  p.out = (float*)d_out; p.ws_all = (char* const*)d_ws_all; p.rank = rank; p.pad = 0;
  hipLaunchKernelGGL(fused_kernel, dim3(NWG), dim3(512), LDS_BYTES, stream, p);
  HIP_CHECK(hipGetLastError());
}
```

```cpp
#include <hip/hip_runtime.h>
#include <rccl/rccl.h>
#include <cstdio>

#define HIP_CHECK(expr) do { hipError_t _e = (expr); if (_e != hipSuccess) { \
    fprintf(stderr, "HIP error %s at %s:%d\n", hipGetErrorString(_e), __FILE__, __LINE__); \
    return; } } while (0)

#define POLL_SLEEP 4
#define FLM 15
#define AHEAD 3
#define MON_SLEEP 2
namespace {

typedef unsigned u32x4 __attribute__((ext_vector_type(4)));
typedef unsigned u32x2 __attribute__((ext_vector_type(2)));
typedef int i32x8 __attribute__((ext_vector_type(8)));
typedef float f32x4 __attribute__((ext_vector_type(4)));
#define LAS __attribute__((address_space(3)))

constexpr int NR = 8;
constexpr int MDIM = 4096, NDIM = 4096, KSH = 512;
constexpr int GEMM0 = 32;
constexpr int NREP = 8;
constexpr int NWG = 256;
constexpr int NTM = 14;
constexpr int A_BYTES = 19 * 2048, B_BYTES = 16 * 2048, BUF_BYTES = A_BYTES + B_BYTES;
constexpr int CTL_OFF = 2 * BUF_BYTES;
constexpr int LDS_BYTES = CTL_OFF + 2048;
constexpr size_t OP_BYTES = 256 * 2048;
constexpr size_t KT_BYTES = 2 * OP_BYTES;
constexpr size_t SHARD_BYTES = 4 * KT_BYTES;
constexpr size_t WS_SRC = 0;
constexpr size_t WS_DST = 2 * SHARD_BYTES;
constexpr size_t WS_READY = WS_DST + NR * SHARD_BYTES;
constexpr size_t WS_CNT = WS_READY + NR * 256 * 4;
constexpr size_t WS_TKT = WS_CNT + 4096;
constexpr size_t WS_PROG = WS_TKT + 1024;
constexpr size_t WS_TICKET = WS_PROG + NREP * 256;
constexpr size_t WS_TMO = WS_TICKET + 16 * 128;
constexpr size_t WS_STAMP = WS_TMO + 1024;
constexpr size_t WS_STAMP2 = WS_STAMP + 256 * 64 * 8;
constexpr size_t WS_END = WS_STAMP2 + 8192 + 256 * 32 * 16;
constexpr unsigned long long DEADLINE = 20000000ull;

struct Params {
  const unsigned char* x; const unsigned char* w; const float* sx; const float* sw; float* out;
  char* const* ws_all; int rank; int pad;
};

#define SB() __builtin_amdgcn_sched_barrier(0)
__device__ __forceinline__ unsigned long long now() { return __builtin_amdgcn_s_memrealtime(); }
__device__ __forceinline__ void st16_sys(void* p, u32x4 v) { asm volatile("global_store_dwordx4 %0, %1, off sc0 sc1\n\ts_nop 1" :: "v"(p), "v"(v) : "memory"); }
__device__ __forceinline__ void st4_sys(void* p, unsigned v) { asm volatile("global_store_dword %0, %1, off sc0 sc1\n\ts_nop 1" :: "v"(p), "v"(v) : "memory"); }
__device__ __forceinline__ unsigned ld4_sys(const void* p) { unsigned v; asm volatile("global_load_dword %0, %1, off sc0 sc1\n\ts_waitcnt vmcnt(0)" : "=&v"(v) : "v"(p) : "memory"); return v; }
__device__ __forceinline__ int tile_start(int pm) { return (pm * 256) / NTM; }

__device__ __forceinline__ bool give_up(unsigned long long t0, unsigned* tmo, unsigned call) {
  if ((unsigned)__builtin_amdgcn_readfirstlane(ld4_sys(tmo)) == call) return true;
  if (now() - t0 > DEADLINE) { st4_sys(tmo, call); return true; }
  return false;
}

#define STAMP(i) do {} while (0)
#define STAMPW(i) do {} while (0)

#define R_B0 "v[200:207]"
#define R_B1 "v[208:215]"
#define R_B2 "v[216:223]"
#define R_B3 "v[224:231]"
#define R_AX "v[232:239]"
#define R_AY "v[240:247]"
#define RD_(f, LO, HI, ALL, addr, OFF) asm volatile("ds_read_b128 " LO ", %1 offset:%c2\n\tds_read_b128 " HI ", %1 offset:%c3" : "={" ALL "}"(f) : "v"(addr), "i"(OFF), "i"((OFF) + 1024))
#define RD_B0(addr, OFF) RD_(r.b0, "v[200:203]", "v[204:207]", R_B0, addr, OFF)
#define RD_B1(addr, OFF) RD_(r.b1, "v[208:211]", "v[212:215]", R_B1, addr, OFF)
#define RD_B2(addr, OFF) RD_(r.b2, "v[216:219]", "v[220:223]", R_B2, addr, OFF)
#define RD_B3(addr, OFF) RD_(r.b3, "v[224:227]", "v[228:231]", R_B3, addr, OFF)
#define RD_AX(addr, OFF) RD_(r.ax, "v[232:235]", "v[236:239]", R_AX, addr, OFF)
#define RD_AY(addr, OFF) RD_(r.ay, "v[240:243]", "v[244:247]", R_AY, addr, OFF)
#define WL(N) asm volatile("s_waitcnt lgkmcnt(" #N ")")
#define MM(ACC, RB, vb, RA, va) asm volatile("v_mfma_f32_16x16x128_f8f6f4 %0, " RB ", " RA ", %0" : "+v"(ACC) : "{" RB "}"(vb), "{" RA "}"(va))
#define MMA4X(m) do { MM(acc[m][0], R_B0, r.b0, R_AX, r.ax); MM(acc[m][1], R_B1, r.b1, R_AX, r.ax); MM(acc[m][2], R_B2, r.b2, R_AX, r.ax); MM(acc[m][3], R_B3, r.b3, R_AX, r.ax); } while (0)
#define MMA4Y(m) do { MM(acc[m][0], R_B0, r.b0, R_AY, r.ay); MM(acc[m][1], R_B1, r.b1, R_AY, r.ay); MM(acc[m][2], R_B2, r.b2, R_AY, r.ay); MM(acc[m][3], R_B3, r.b3, R_AY, r.ay); } while (0)
struct Regs { i32x8 b0, b1, b2, b3, ax, ay; };
__device__ __forceinline__ void prime_reads(Regs& r, unsigned pa, unsigned pb) {
  RD_AX(pa, 0); RD_B0(pb, 0); RD_B1(pb, 2048); RD_B2(pb, 4096); RD_B3(pb, 6144); RD_AY(pa, 2048);
}
__device__ __forceinline__ void dma(const char* sbase, unsigned voff, unsigned ldsa) {
  asm volatile("s_mov_b32 m0, %2\n\ts_nop 0\n\tglobal_load_lds_dwordx4 %0, %1 sc0 sc1" :: "v"(voff), "s"(sbase), "s"(ldsa) : "memory");
}
__device__ __forceinline__ void stage_all(const char* ga, const char* gb, unsigned voff, unsigned nl, bool a4) {
  const unsigned la = nl, lb = nl + A_BYTES;
  dma(gb, voff, lb); dma(gb + 8192, voff, lb + 8192); dma(gb + 2 * 8192, voff, lb + 2 * 8192); dma(gb + 3 * 8192, voff, lb + 3 * 8192);
  dma(ga, voff, la); dma(ga + 8192, voff, la + 8192); dma(ga + 2 * 8192, voff, la + 2 * 8192); dma(ga + 3 * 8192, voff, la + 3 * 8192);
  if (a4) dma(ga + 4 * 8192, voff, la + 4 * 8192);
}
__device__ __forceinline__ float silu(float acc, float s) {
  const float y = acc * s;
  const float c = __builtin_amdgcn_fmed3f(y, -60.0f, 60.0f);
  const float e = __builtin_amdgcn_exp2f(c * -1.44269504088896340736f);
  return y * __builtin_amdgcn_rcpf(1.0f + e);
}

__device__ __forceinline__ unsigned chunk_off(int q, int pi) {
  int p, op, i;
  if (q < 16) { p = 0; op = 1; i = q; }
  else if (q >= 112) { p = 3; op = 0; i = q - 112; }
  else { const int j = q - 16; op = j & 1; p = (j >> 5) + op; i = (j & 31) >> 1; }
  return (unsigned)(p * KT_BYTES + op * OP_BYTES + ((i + 2 * pi) & 15) * 32768);
}
__device__ __forceinline__ int chunk_pos(int p, int op, int c, int pi) {
  const int i = (c - 2 * pi) & 15;
  if (op) return p == 0 ? i : 16 + 32 * (p - 1) + 2 * i + 1;
  return p == 3 ? 112 + i : 16 + 32 * p + 2 * i;
}

__global__ __launch_bounds__(512, 2) void fused_kernel(Params P) {
  extern __shared__ __attribute__((aligned(16))) unsigned char smem[];
  LAS unsigned char* lds = (LAS unsigned char*)smem;
  LAS unsigned* ctl = (LAS unsigned*)(lds + CTL_OFF);
  const int tid = threadIdx.x, lane = tid & 63, wid = __builtin_amdgcn_readfirstlane(tid >> 6);
  const int b = blockIdx.x, rank = P.rank;
  char* myws = P.ws_all[rank];
  unsigned* tmo = (unsigned*)(myws + WS_TMO);
  STAMP(0);

  unsigned ticket = 0;
  if (tid == 448) ticket = __hip_atomic_fetch_add((unsigned*)(myws + WS_TICKET + (b & 15) * 128), 1u, __ATOMIC_RELAXED, __HIP_MEMORY_SCOPE_AGENT);
  {
    const int p = b >> 6, G = b & 63;
    u32x4 o[4];
    if (wid < 2) {
      const int c = lane & 15, q = lane >> 4;
      const unsigned char* src = P.w + (size_t)(p * 128 + wid * 64 + q * 16) * NDIM + G * 64 + c * 4;
      unsigned d[16];
#pragma unroll
      for (int j = 0; j < 16; ++j) d[j] = *(const unsigned*)(src + (size_t)j * NDIM);
#pragma unroll
      for (int jj = 0; jj < 4; ++jj) {
        const unsigned a0 = d[4 * jj], a1 = d[4 * jj + 1], a2 = d[4 * jj + 2], a3 = d[4 * jj + 3];
        const unsigned t0 = __builtin_amdgcn_perm(a1, a0, 0x05010400u), t1 = __builtin_amdgcn_perm(a1, a0, 0x07030602u);
        const unsigned t2 = __builtin_amdgcn_perm(a3, a2, 0x05010400u), t3 = __builtin_amdgcn_perm(a3, a2, 0x07030602u);
        o[0][jj] = __builtin_amdgcn_perm(t2, t0, 0x05040100u); o[1][jj] = __builtin_amdgcn_perm(t2, t0, 0x07060302u);
        o[2][jj] = __builtin_amdgcn_perm(t3, t1, 0x05040100u); o[3][jj] = __builtin_amdgcn_perm(t3, t1, 0x07060302u);
      }
    } else if (wid < 6) {
      const int fr = lane & 15, fq = lane >> 4;
      const unsigned char* src = P.x + (size_t)((4 * G + wid - 2) * 16 + fr) * KSH + p * 128 + fq * 16;
      o[0] = *(const u32x4*)src; o[1] = *(const u32x4*)(src + 64);
    }
    if (tid == 448) ctl[4] = ticket / 16 + 1;
    __syncthreads();
    const unsigned call_ = ctl[4];
    char* dst = myws + WS_SRC + (size_t)(call_ & 1) * SHARD_BYTES + (size_t)p * KT_BYTES;
    if (wid < 2) {
#pragma unroll
      for (int i = 0; i < 4; ++i) st16_sys(dst + OP_BYTES + (size_t)(4 * G + i) * 2048 + wid * 1024 + lane * 16, o[i]);
    } else if (wid < 6) {
      char* d2 = dst + (size_t)(4 * G + wid - 2) * 2048 + lane * 16;
      st16_sys(d2, o[0]); st16_sys(d2 + 1024, o[1]);
    }
    asm volatile("s_waitcnt vmcnt(0)\n\ts_barrier" ::: "memory");
    if (tid < NR) st4_sys(P.ws_all[tid] + WS_READY + (size_t)(rank * 256 + b) * 4, call_);
  }
  const unsigned call = (unsigned)__builtin_amdgcn_readfirstlane(ctl[4]);
  STAMP(1);

  if (b < GEMM0) {
    for (int i = tid; i < 320; i += 512) ctl[8 + i] = 0;
    asm volatile("s_waitcnt lgkmcnt(0)\n\ts_barrier" ::: "memory");
    const unsigned lctl = (unsigned)(size_t)lds + CTL_OFF;
    if (b >= 28) {
      if (b != 28 || wid != 7) return;
      const unsigned pbase = call * 256;
      const int mp = lane >> 3, mo = lane & 7;
      unsigned* pl = (unsigned*)(myws + WS_PROG + mo * 256) + mp;
      const unsigned long long t0 = now();
      {
        const unsigned* fl = (const unsigned*)(myws + WS_READY) + rank * 256 + lane * 4;
        for (;;) {
          u32x4 v; asm volatile("global_load_dwordx4 %0, %1, off sc0 sc1\n\ts_waitcnt vmcnt(0)" : "=&v"(v) : "v"(fl) : "memory");
          if (__all(v[0] >= call && v[1] >= call && v[2] >= call && v[3] >= call)) break;
          if (give_up(t0, tmo, call)) break;
          __builtin_amdgcn_s_sleep(2);
        }
        if (mp == 7) st4_sys(pl, pbase + 128);
      }
      const unsigned* cnt = (const unsigned*)(myws + WS_CNT) + (mp < 7 ? mp : 0) * 128;
      int pos = mp < 7 ? 0 : 128;
      for (;;) {
        const int q = pos + mo < 128 ? pos + mo : 127;
        unsigned v; asm volatile("global_load_dword %0, %1, off sc1\n\ts_waitcnt vmcnt(0)" : "=&v"(v) : "v"(cnt + q) : "memory");
        const unsigned long long m = __ballot(mp < 7 && pos + mo < 128 && v >= call);
        const int adv = __builtin_ctz(~(unsigned)((m >> (mp * 8)) & 0xFF));
        if (adv > 0) { pos += adv; st4_sys(pl, pbase + pos); }
        if (__all(pos >= 128)) { STAMPW(40); break; }
        if (m == 0) { if (give_up(t0, tmo, call)) break; __builtin_amdgcn_s_sleep(MON_SLEEP); }
      }
      return;
    }
    const int pi = b % 7, peer = (rank + 1 + pi) & 7;
    auto lds_rd = [&](unsigned a) -> unsigned { unsigned v; asm volatile("ds_read_b32 %0, %1\n\ts_waitcnt lgkmcnt(0)" : "=v"(v) : "v"(a) : "memory"); return (unsigned)__builtin_amdgcn_readfirstlane(v); };
    if (wid == 7) {
      unsigned* tk = (unsigned*)(myws + WS_TKT + pi * 128);
      const unsigned base = (call - 1) * 132;
      const unsigned long long t0 = now();
      for (int t = 0;; ++t) {
        if (t >= AHEAD) while (lds_rd(lctl + 4 * (16 + t - AHEAD)) != 7 && now() - t0 < 2 * DEADLINE) __builtin_amdgcn_s_sleep(1);
        unsigned v = 0;
        if (lane == 0) asm volatile("global_atomic_add %0, %1, %2, off sc0 sc1\n\ts_waitcnt vmcnt(0)" : "=&v"(v) : "v"(tk), "v"(1u) : "memory");
        v = (unsigned)__builtin_amdgcn_readfirstlane(v) - base;
        const bool stop = v >= 128 || t >= 127;
        if (lane == 0) asm volatile("ds_write_b32 %0, %1" :: "v"(lctl + 4 * (160 + t)), "v"(stop ? 255u : v + 1) : "memory");
        if (stop) break;
      }
      return;
    }
    if (wid == 0) {
      const unsigned* fl = (const unsigned*)(myws + WS_READY) + peer * 256 + lane * 4;
      const unsigned long long t0 = now();
      for (;;) {
        u32x4 v; asm volatile("global_load_dwordx4 %0, %1, off sc0 sc1\n\ts_waitcnt vmcnt(0)" : "=&v"(v) : "v"(fl) : "memory");
        if (__all(v[0] >= call && v[1] >= call && v[2] >= call && v[3] >= call)) break;
        if (give_up(t0, tmo, call)) break;
        __builtin_amdgcn_s_sleep(4);
      }
      if (lane == 0) asm volatile("ds_write_b32 %0, %1" :: "v"(lctl + 4 * 8), "v"(1u) : "memory");
    }
    { const unsigned long long t0 = now(); while (lds_rd(lctl + 4 * 8) == 0 && now() - t0 < 2 * DEADLINE) __builtin_amdgcn_s_sleep(2); }
    const char* srcb = P.ws_all[peer] + WS_SRC + (size_t)(call & 1) * SHARD_BYTES;
    char* dstb = myws + WS_DST + (size_t)peer * SHARD_BYTES;
    unsigned* cntb = (unsigned*)(myws + WS_CNT) + pi * 128;
    const unsigned voff = wid * 1024 + lane * 16;
    const bool five = wid < 4;
    auto get_pos = [&](int t) -> int {
      unsigned v; const unsigned long long t0 = now();
      while ((v = lds_rd(lctl + 4 * (160 + t))) == 0) { if (now() - t0 > 2 * DEADLINE) { v = 255; break; } __builtin_amdgcn_s_sleep(1); }
      return v == 255 ? -1 : (int)v - 1;
    };
    typedef int i32x12 __attribute__((ext_vector_type(12)));
    i32x8 ra; i32x12 rb;
    int q = get_pos(0);
    STAMPW(20 + wid);
    if (q >= 0) {
      unsigned o = chunk_off(q, pi);
      asm volatile("global_load_dwordx4 v[200:203], %1, %2 sc0 sc1\n\tglobal_load_dwordx4 v[204:207], %1, %3 sc0 sc1\n\ts_waitcnt vmcnt(0)"
                   : "={v[200:207]}"(ra) : "v"(voff), "s"(srcb + o), "s"(srcb + o + 7168) : "memory");
      int qprev = -1;
      auto arrive = [&](int t, int qq) {
        unsigned old = 0;
        if (lane == 0) asm volatile("ds_add_rtn_u32 %0, %1, %2\n\ts_waitcnt lgkmcnt(0)" : "=&v"(old) : "v"(lctl + 4 * (16 + t)), "v"(1u) : "memory");
        if (__builtin_amdgcn_readfirstlane(old) == 6 && lane == 0) st4_sys(cntb + qq, call);
      };
      int t = 0;
      for (;; ++t) {
        const char* s = srcb + o; char* d = dstb + o;
        if (five)
          asm volatile("global_load_dwordx4 v[208:211], %1, %2 sc0 sc1\n\tglobal_load_dwordx4 v[212:215], %1, %3 sc0 sc1\n\tglobal_load_dwordx4 v[216:219], %1, %4 sc0 sc1\n\t"
                       "global_store_dwordx4 %1, v[200:203], %5 sc0 sc1\n\tglobal_store_dwordx4 %1, v[204:207], %6 sc0 sc1\n\ts_waitcnt vmcnt(2)"
                       : "={v[208:219]}"(rb) : "v"(voff), "s"(s + 2 * 7168), "s"(s + 3 * 7168), "s"(s + 4 * 7168), "s"(d), "s"(d + 7168), "{v[200:207]}"(ra) : "memory");
        else
          asm volatile("global_load_dwordx4 v[208:211], %1, %2 sc0 sc1\n\tglobal_load_dwordx4 v[212:215], %1, %3 sc0 sc1\n\t"
                       "global_store_dwordx4 %1, v[200:203], %4 sc0 sc1\n\tglobal_store_dwordx4 %1, v[204:207], %5 sc0 sc1\n\ts_waitcnt vmcnt(2)"
                       : "={v[208:219]}"(rb) : "v"(voff), "s"(s + 2 * 7168), "s"(s + 3 * 7168), "s"(d), "s"(d + 7168), "{v[200:207]}"(ra) : "memory");
        if (t > 0) arrive(t - 1, qprev);
        qprev = q;
        const int qn = get_pos(t + 1);
        if (qn >= 0) o = chunk_off(qn, pi);
        const char* sn = srcb + o;
        if (five)
          asm volatile("global_load_dwordx4 v[200:203], %1, %2 sc0 sc1\n\tglobal_load_dwordx4 v[204:207], %1, %3 sc0 sc1\n\t"
                       "global_store_dwordx4 %1, v[208:211], %4 sc0 sc1\n\tglobal_store_dwordx4 %1, v[212:215], %5 sc0 sc1\n\tglobal_store_dwordx4 %1, v[216:219], %6 sc0 sc1\n\ts_waitcnt vmcnt(3)"
                       : "={v[200:207]}"(ra) : "v"(voff), "s"(sn), "s"(sn + 7168), "s"(d + 2 * 7168), "s"(d + 3 * 7168), "s"(d + 4 * 7168), "{v[208:219]}"(rb) : "memory");
        else
          asm volatile("global_load_dwordx4 v[200:203], %1, %2 sc0 sc1\n\tglobal_load_dwordx4 v[204:207], %1, %3 sc0 sc1\n\t"
                       "global_store_dwordx4 %1, v[208:211], %4 sc0 sc1\n\tglobal_store_dwordx4 %1, v[212:215], %5 sc0 sc1\n\ts_waitcnt vmcnt(2)"
                       : "={v[200:207]}"(ra) : "v"(voff), "s"(sn), "s"(sn + 7168), "s"(d + 2 * 7168), "s"(d + 3 * 7168), "{v[208:219]}"(rb) : "memory");
        if (qn < 0) break;
        q = qn;
      }
      asm volatile("s_waitcnt vmcnt(0)" ::: "memory");
      arrive(t, qprev);
    }
    STAMPW(10 + wid);
    return;
  }

  const int g = b - GEMM0, xcd = g & 7, idx = g >> 3;
  const int pm = (xcd & 1) * 7 + idx % 7, pn = (xcd >> 1) * 4 + idx / 7;
  const int g0 = tile_start(pm), g1 = tile_start(pm + 1), wr = wid >> 2, wc = wid & 3;
  const bool a4 = wid < (g1 - g0) * 2 - 32;
  const char* own = myws + WS_SRC + (size_t)(call & 1) * SHARD_BYTES;
  const char* dstb = myws + WS_DST;
  const size_t aoff = (size_t)g0 * 2048, boff = OP_BYTES + (size_t)pn * B_BYTES;
  auto kt_base = [&](int kt) -> const char* { const int s = kt >> 2, p = kt & 3; return (s == rank ? own : dstb + (size_t)s * SHARD_BYTES) + (size_t)p * KT_BYTES; };

  unsigned pollA, thr;
  {
    const int kt = lane & 31, s = kt >> 2, p = kt & 3, pi = (s - rank - 1) & 7;
    const int qa = chunk_pos(p, 0, g0 >> 4, pi), qb = chunk_pos(p, 0, (g1 - 1) >> 4, pi), qc = chunk_pos(p, 1, pn, pi);
    pollA = (unsigned)WS_PROG + (unsigned)(g & (NREP - 1)) * 256 + pi * 4;
    thr = call * 256 + (pi == 7 ? 128 : max(max(qa, qb), qc) + 1);
  }

  f32x4 acc[10][4];
#pragma unroll
  for (int m = 0; m < 10; ++m)
#pragma unroll
    for (int n = 0; n < 4; ++n) acc[m][n] = (f32x4){0.f, 0.f, 0.f, 0.f};

  const unsigned lbase = (unsigned)(size_t)lds;
  const unsigned voff = wid * 1024 + lane * 16;
  unsigned pa = lbase + wr * (10 * 2048) + lane * 16, pb = lbase + A_BYTES + wc * (4 * 2048) + lane * 16;
  unsigned npa = pa + BUF_BYTES, npb = pb + BUF_BYTES;
  const unsigned ctl_addr = lbase + CTL_OFF;
  unsigned ready = 0, taken = 0;
  auto wait_tile = [&]() {
    if (wid == 0) {
      const unsigned long long t0 = now();
      for (;;) {
        unsigned q; asm volatile("global_load_dword %0, %1, %2 sc1\n\ts_waitcnt vmcnt(0)" : "=&v"(q) : "v"(pollA), "s"(myws) : "memory");
        unsigned m = (unsigned)__ballot(q >= thr);
        if ((m & ~taken) == 0 && give_up(t0, tmo, call)) m = ~0u;
        if ((m & ~taken) != 0) { if (lane == 0) asm volatile("ds_write_b32 %0, %1" :: "v"(ctl_addr + 8), "v"(m) : "memory"); break; }
        __builtin_amdgcn_s_sleep(POLL_SLEEP);
      }
    }
    unsigned mr;
    asm volatile("s_waitcnt lgkmcnt(0)\n\ts_barrier\n\tds_read_b32 %0, %1\n\ts_waitcnt lgkmcnt(0)" : "=v"(mr) : "v"(ctl_addr + 8) : "memory");
    ready = (unsigned)__builtin_amdgcn_readfirstlane(mr);
  };
  wait_tile();
  STAMP(2);
  Regs r;
  { const int k = __builtin_ctz(ready); taken |= 1u << k; const char* kb = kt_base(k);
    stage_all(kb + aoff, kb + boff, voff, lbase + wid * 1024, a4);
    asm volatile("s_waitcnt vmcnt(0)\n\ts_barrier" ::: "memory");
    prime_reads(r, pa, pb); }
  int cur = 0;
  for (int n = 0; n < 32; ++n) {
    const unsigned nl = lbase + (cur ^ 1) * BUF_BYTES + wid * 1024;
    const unsigned la = nl, lb = nl + A_BYTES;
    unsigned q0;
    const unsigned ctl_cur = ctl_addr + (n & 1) * 4;
#define DB(i) dma(gb + (i) * 8192, voff, lb + (i) * 8192)
#define DA(i) dma(ga + (i) * 8192, voff, la + (i) * 8192)
    auto try_stage = [&](int path) -> bool {
      const unsigned cand = ready & ~taken;
      if (cand == 0) return false;
      const int k = __builtin_ctz(cand); taken |= 1u << k;
      if (wr == 1) {
        const char* kb = kt_base(k);
        const char* ga = kb + aoff; const char* gb = kb + boff;
        const unsigned vo2 = voff - 4096, la2 = la - 4096, lb2 = lb - 4096;
        dma(gb, vo2, lb2); DB(0); dma(gb + 8192, vo2, lb2 + 8192); DB(1); dma(gb + 2 * 8192, vo2, lb2 + 2 * 8192); DB(2); dma(gb + 3 * 8192, vo2, lb2 + 3 * 8192); DB(3);
        dma(ga, vo2, la2); DA(0); dma(ga + 8192, vo2, la2 + 8192); DA(1); dma(ga + 2 * 8192, vo2, la2 + 2 * 8192); DA(2); dma(ga + 3 * 8192, vo2, la2 + 3 * 8192); DA(3);
        dma(ga + 4 * 8192, vo2, la2 + 4 * 8192); if (a4) DA(4);
      }
      return true;
    };
#define POLL_ISSUE() do { if (wid == 0) asm volatile("global_load_dword %0, %1, %2 sc1" : "=&v"(q0) : "v"(pollA), "s"(myws) : "memory"); } while (0)
#define POLL_PUBLISH(addr) do { if (wid == 0) {   \
      asm volatile("s_waitcnt vmcnt(0)" : "+v"(q0) :: "memory"); \
      const unsigned m_ = (unsigned)__ballot(q0 >= thr); \
      if (lane == 0) asm volatile("ds_write_b32 %0, %1" :: "v"(addr), "v"(m_) : "memory"); } } while (0)
    POLL_ISSUE();
    bool stage = try_stage(0);
    const bool mid = !stage && n < 31;
    __builtin_amdgcn_s_setprio(1);
    WL(8); MM(acc[0][0], R_B0, r.b0, R_AX, r.ax);
    WL(6); MM(acc[0][1], R_B1, r.b1, R_AX, r.ax);
    WL(4); MM(acc[0][2], R_B2, r.b2, R_AX, r.ax);
    WL(2); MM(acc[0][3], R_B3, r.b3, R_AX, r.ax);
    RD_AX(pa, 2 * 2048);
    WL(2); MMA4Y(1); RD_AY(pa, 3 * 2048);
    WL(2); MMA4X(2); RD_AX(pa, 4 * 2048);
    WL(2); MMA4Y(3); RD_AY(pa, 5 * 2048);
    WL(2); MMA4X(4); RD_AX(pa, 6 * 2048);
    WL(2); MMA4Y(5); RD_AY(pa, 7 * 2048);
    if (mid) {
      POLL_PUBLISH(ctl_addr + 12);
      unsigned mm;
      asm volatile("s_waitcnt lgkmcnt(0)\n\ts_barrier\n\tds_read_b32 %0, %1\n\ts_waitcnt lgkmcnt(0)" : "=v"(mm) : "v"(ctl_addr + 12) : "memory");
      ready = (unsigned)__builtin_amdgcn_readfirstlane(mm);
      POLL_ISSUE();
      stage = try_stage(1);
    }
    WL(2); MMA4X(6); RD_AX(pa, 8 * 2048);
    if (!mid) POLL_PUBLISH(ctl_cur);
    WL(2); MMA4Y(7); RD_AY(pa, 9 * 2048);
    WL(2); MMA4X(8);
    if (mid) POLL_PUBLISH(ctl_cur);
    asm volatile("s_waitcnt vmcnt(0) lgkmcnt(0)\n\ts_barrier" ::: "memory");
    unsigned mreg;
    asm volatile("ds_read_b32 %0, %1" : "=v"(mreg) : "v"(ctl_cur));
    RD_AX(npa, 0);
    asm volatile(
      "s_cmp_lg_u32 %[wr], 0\n\t"
      "s_cbranch_scc1 1f\n\t"
      "v_mfma_f32_16x16x128_f8f6f4 %[c0], v[200:207], v[240:247], %[c0]\n\t"
      "ds_read_b128 v[200:203], %[pb] offset:0\n\tds_read_b128 v[204:207], %[pb] offset:1024\n\t"
      "v_mfma_f32_16x16x128_f8f6f4 %[c1], v[208:215], v[240:247], %[c1]\n\t"
      "ds_read_b128 v[208:211], %[pb] offset:2048\n\tds_read_b128 v[212:215], %[pb] offset:3072\n\t"
      "v_mfma_f32_16x16x128_f8f6f4 %[c2], v[216:223], v[240:247], %[c2]\n\t"
      "ds_read_b128 v[216:219], %[pb] offset:4096\n\tds_read_b128 v[220:223], %[pb] offset:5120\n\t"
      "v_mfma_f32_16x16x128_f8f6f4 %[c3], v[224:231], v[240:247], %[c3]\n\t"
      "ds_read_b128 v[224:227], %[pb] offset:6144\n\tds_read_b128 v[228:231], %[pb] offset:7168\n\t"
      "s_branch 2f\n"
      "1:\n\t"
      "ds_read_b128 v[200:203], %[pb] offset:0\n\tds_read_b128 v[204:207], %[pb] offset:1024\n\t"
      "ds_read_b128 v[208:211], %[pb] offset:2048\n\tds_read_b128 v[212:215], %[pb] offset:3072\n\t"
      "ds_read_b128 v[216:219], %[pb] offset:4096\n\tds_read_b128 v[220:223], %[pb] offset:5120\n\t"
      "ds_read_b128 v[224:227], %[pb] offset:6144\n\tds_read_b128 v[228:231], %[pb] offset:7168\n"
      "2:"
      : [c0] "+v"(acc[9][0]), [c1] "+v"(acc[9][1]), [c2] "+v"(acc[9][2]), [c3] "+v"(acc[9][3]),
        "+{v[200:207]}"(r.b0), "+{v[208:215]}"(r.b1), "+{v[216:223]}"(r.b2), "+{v[224:231]}"(r.b3)
      : [wr] "s"(wr), [pb] "v"(npb), "{v[240:247]}"(r.ay) : "scc");
    RD_AY(npa, 2048);
    __builtin_amdgcn_s_setprio(0);
    asm volatile("s_waitcnt lgkmcnt(12)" : "+v"(mreg));
    ready = (unsigned)__builtin_amdgcn_readfirstlane(mreg);
#undef DA
#undef DB
#undef POLL_ISSUE
#undef POLL_PUBLISH
    if (n == 31) break;
    if (!stage) {
      if ((ready & ~taken) == 0) wait_tile();
      const int k2 = __builtin_ctz(ready & ~taken); taken |= 1u << k2;
      const char* kb2 = kt_base(k2);
      stage_all(kb2 + aoff, kb2 + boff, voff, nl, a4);
      asm volatile("s_waitcnt vmcnt(0)\n\ts_barrier" ::: "memory");
      prime_reads(r, npa, npb);
    }
    { unsigned t = pa; pa = npa; npa = t; t = pb; pb = npb; npb = t; cur ^= 1; }
  }
  asm volatile("s_waitcnt lgkmcnt(0)\n\ts_nop 15\n\ts_nop 15\n\ts_barrier" ::: "memory");
  STAMP(3);

  {
    float s = P.sx[0] * P.sw[0];
    const bool bad = (unsigned)__builtin_amdgcn_readfirstlane(ld4_sys(tmo)) == call;
    if (bad) s = __builtin_nanf("");
    LAS unsigned char* reg = lds + wid * (2 * 16 * 272);
    const int fr = lane & 15, fq = lane >> 4;
#pragma unroll
    for (int m = 0; m < 10; ++m) {
      const int gg = g0 + wr * 10 + m;
      if (gg < g1 && (wr == 0 || m < 9)) {
        LAS unsigned char* rr = reg + (m & 1) * (16 * 272);
#pragma unroll
        for (int j = 0; j < 4; ++j) {
          f32x4 v = {silu(acc[m][0][j], s), silu(acc[m][1][j], s), silu(acc[m][2][j], s), silu(acc[m][3][j], s)};
          *(LAS f32x4*)(rr + fr * 272 + fq * 64 + j * 16) = v;
        }
        float* o = P.out + (size_t)(gg * 16 + fq) * NDIM + pn * 256 + wc * 64 + fr * 4;
#pragma unroll
        for (int r4 = 0; r4 < 4; ++r4) {
          f32x4 v = *(const LAS f32x4*)(rr + (r4 * 4 + fq) * 272 + fr * 16);
          *(f32x4*)(o + (size_t)r4 * 4 * NDIM) = v;
        }
      }
    }
  }
}

}

extern "C" void kernel_launch(int rank, int nranks, ncclComm_t comm, hipStream_t stream,
                              void* const* d_in, const int* in_sizes, int n_in,
                              void* d_out, int out_size, const int* in_dtypes, int out_dtype,
                              void* const* d_out_all, void* const* d_ws_all, size_t ws_bytes) {
  (void)comm; (void)in_sizes; (void)n_in; (void)out_size; (void)in_dtypes; (void)out_dtype; (void)d_out_all;
  if (nranks != NR || (unsigned)rank >= (unsigned)NR || ws_bytes < WS_END) { fprintf(stderr, "unsupported configuration\n"); return; }
  static bool attr_done[NR] = {};
  if (!attr_done[rank]) {
    HIP_CHECK(hipFuncSetAttribute((const void*)fused_kernel, hipFuncAttributeMaxDynamicSharedMemorySize, LDS_BYTES));
    attr_done[rank] = true;
  }
  Params p;
  p.x = (const unsigned char*)d_in[0]; p.w = (const unsigned char*)d_in[1]; p.sx = (const float*)d_in[2]; p.sw = (const float*)d_in[3];
  p.out = (float*)d_out; p.ws_all = (char* const*)d_ws_all; p.rank = rank; p.pad = 0;
  hipLaunchKernelGGL(fused_kernel, dim3(NWG), dim3(512), LDS_BYTES, stream, p);
  HIP_CHECK(hipGetLastError());
}
```

```cpp
#include <hip/hip_runtime.h>
#include <rccl/rccl.h>
#include <cstdio>

#define HIP_CHECK(expr) do { hipError_t _e = (expr); if (_e != hipSuccess) { \
    fprintf(stderr, "HIP error %s at %s:%d\n", hipGetErrorString(_e), __FILE__, __LINE__); \
    return; } } while (0)

#define POLL_SLEEP 4
#define FLM 15
#define AHEAD 4
#define MON_SLEEP 2
namespace {

typedef unsigned u32x4 __attribute__((ext_vector_type(4)));
typedef unsigned u32x2 __attribute__((ext_vector_type(2)));
typedef int i32x8 __attribute__((ext_vector_type(8)));
typedef float f32x4 __attribute__((ext_vector_type(4)));
#define LAS __attribute__((address_space(3)))

constexpr int NR = 8;
constexpr int MDIM = 4096, NDIM = 4096, KSH = 512;
constexpr int GEMM0 = 32;
constexpr int NREP = 8;
constexpr int NWG = 256;
constexpr int NTM = 14;
constexpr int A_BYTES = 19 * 2048, B_BYTES = 16 * 2048, BUF_BYTES = A_BYTES + B_BYTES;
constexpr int CTL_OFF = 2 * BUF_BYTES;
constexpr int LDS_BYTES = CTL_OFF + 2048;
constexpr size_t OP_BYTES = 256 * 2048;
constexpr size_t KT_BYTES = 2 * OP_BYTES;
constexpr size_t SHARD_BYTES = 4 * KT_BYTES;
constexpr size_t SRC_RAW = 0, SRC_PACK = SHARD_BYTES, SRC_EXC = SRC_PACK + 2048 * 1792, SRC_HDR = SRC_EXC + 1024 * 256, SRC_BYTES = 2 * SHARD_BYTES;
static_assert(SRC_HDR + 4096 <= SRC_BYTES, "SRC layout");
constexpr size_t WS_SRC = 0;
constexpr size_t WS_DST = 2 * SRC_BYTES;
constexpr size_t WS_READY = WS_DST + NR * SHARD_BYTES;
constexpr size_t WS_READY2 = WS_READY + NR * 256 * 4;
constexpr size_t WS_CNT = WS_READY2 + NR * 256 * 4;
constexpr size_t WS_TKT = WS_CNT + 4096;
constexpr size_t WS_PROG = WS_TKT + 1024;
constexpr size_t WS_TICKET = WS_PROG + NREP * 256;
constexpr size_t WS_TMO = WS_TICKET + 16 * 128;
constexpr size_t WS_STAMP = WS_TMO + 1024;
constexpr size_t WS_STAMP2 = WS_STAMP + 256 * 64 * 8;
constexpr size_t WS_END = WS_STAMP2 + 8192 + 256 * 32 * 16 + 32 * 24 * 8;
constexpr unsigned long long DEADLINE = 20000000ull;

struct Params {
  const unsigned char* x; const unsigned char* w; const float* sx; const float* sw; float* out;
  char* const* ws_all; int rank; int pad;
};

#define SB() __builtin_amdgcn_sched_barrier(0)
#define EXP_LOCALEXC 0
#define UNI(x) ((unsigned)__builtin_amdgcn_readfirstlane((int)(x)))
__device__ __forceinline__ unsigned long long now() { return __builtin_amdgcn_s_memrealtime(); }
__device__ __forceinline__ void st16_sys(void* p, u32x4 v) { asm volatile("global_store_dwordx4 %0, %1, off sc0 sc1\n\ts_nop 1" :: "v"(p), "v"(v) : "memory"); }
__device__ __forceinline__ void st4_sys(void* p, unsigned v) { asm volatile("global_store_dword %0, %1, off sc0 sc1\n\ts_nop 1" :: "v"(p), "v"(v) : "memory"); }
__device__ __forceinline__ unsigned ld4_sys(const void* p) { unsigned v; asm volatile("global_load_dword %0, %1, off sc0 sc1\n\ts_waitcnt vmcnt(0)" : "=&v"(v) : "v"(p) : "memory"); return v; }
__device__ __forceinline__ int tile_start(int pm) { return (pm * 256) / NTM; }

__device__ __forceinline__ bool give_up(unsigned long long t0, unsigned* tmo, unsigned call) {
  if ((unsigned)__builtin_amdgcn_readfirstlane(ld4_sys(tmo)) == call) return true;
  if (now() - t0 > DEADLINE) { st4_sys(tmo, call); return true; }
  return false;
}

#define STAMP(i) do {} while (0)
#define STAMPW(i) do {} while (0)

#define R_B0 "v[200:207]"
#define R_B1 "v[208:215]"
#define R_B2 "v[216:223]"
#define R_B3 "v[224:231]"
#define R_AX "v[232:239]"
#define R_AY "v[240:247]"
#define R_AZ "v[248:255]"
#define RD_(f, LO, HI, ALL, addr, OFF) asm volatile("ds_read_b128 " LO ", %1 offset:%c2\n\tds_read_b128 " HI ", %1 offset:%c3" : "={" ALL "}"(f) : "v"(addr), "i"(OFF), "i"((OFF) + 1024))
#define RD_B0(addr, OFF) RD_(r.b0, "v[200:203]", "v[204:207]", R_B0, addr, OFF)
#define RD_B1(addr, OFF) RD_(r.b1, "v[208:211]", "v[212:215]", R_B1, addr, OFF)
#define RD_B2(addr, OFF) RD_(r.b2, "v[216:219]", "v[220:223]", R_B2, addr, OFF)
#define RD_B3(addr, OFF) RD_(r.b3, "v[224:227]", "v[228:231]", R_B3, addr, OFF)
#define RD_AX(addr, OFF) RD_(r.ax, "v[232:235]", "v[236:239]", R_AX, addr, OFF)
#define RD_AY(addr, OFF) RD_(r.ay, "v[240:243]", "v[244:247]", R_AY, addr, OFF)
#define RD_AZ(addr, OFF) RD_(r.az, "v[248:251]", "v[252:255]", R_AZ, addr, OFF)
#define WL(N) asm volatile("s_waitcnt lgkmcnt(" #N ")")
#define MM(ACC, RB, vb, RA, va) asm volatile("v_mfma_f32_16x16x128_f8f6f4 %0, " RB ", " RA ", %0" : "+v"(ACC) : "{" RB "}"(vb), "{" RA "}"(va))
#define MMA4X(m) do { MM(acc[m][0], R_B0, r.b0, R_AX, r.ax); MM(acc[m][1], R_B1, r.b1, R_AX, r.ax); MM(acc[m][2], R_B2, r.b2, R_AX, r.ax); MM(acc[m][3], R_B3, r.b3, R_AX, r.ax); } while (0)
#define MMA4Y(m) do { MM(acc[m][0], R_B0, r.b0, R_AY, r.ay); MM(acc[m][1], R_B1, r.b1, R_AY, r.ay); MM(acc[m][2], R_B2, r.b2, R_AY, r.ay); MM(acc[m][3], R_B3, r.b3, R_AY, r.ay); } while (0)
#define MMA4Z(m) do { MM(acc[m][0], R_B0, r.b0, R_AZ, r.az); MM(acc[m][1], R_B1, r.b1, R_AZ, r.az); MM(acc[m][2], R_B2, r.b2, R_AZ, r.az); MM(acc[m][3], R_B3, r.b3, R_AZ, r.az); } while (0)
struct Regs { i32x8 b0, b1, b2, b3, ax, ay, az; };
__device__ __forceinline__ void prime_reads(Regs& r, unsigned pa, unsigned pb) {
  RD_AX(pa, 0); RD_B0(pb, 0); RD_B1(pb, 2048); RD_B2(pb, 4096); RD_B3(pb, 6144); RD_AY(pa, 2048); RD_AZ(pa, 2 * 2048);
}
__device__ __forceinline__ void dma(const char* sbase, unsigned voff, unsigned ldsa) {
  asm volatile("s_mov_b32 m0, %2\n\ts_nop 0\n\tglobal_load_lds_dwordx4 %0, %1 sc0 sc1" :: "v"(voff), "s"(sbase), "s"(ldsa) : "memory");
}
__device__ __forceinline__ void stage_all(const char* ga, const char* gb, unsigned voff, unsigned nl, bool a4) {
  const unsigned la = nl, lb = nl + A_BYTES;
  dma(gb, voff, lb); dma(gb + 8192, voff, lb + 8192); dma(gb + 2 * 8192, voff, lb + 2 * 8192); dma(gb + 3 * 8192, voff, lb + 3 * 8192);
  dma(ga, voff, la); dma(ga + 8192, voff, la + 8192); dma(ga + 2 * 8192, voff, la + 2 * 8192); dma(ga + 3 * 8192, voff, la + 3 * 8192);
  if (a4) dma(ga + 4 * 8192, voff, la + 4 * 8192);
}
__device__ __forceinline__ float silu(float acc, float s) {
  const float y = acc * s;
  const float c = __builtin_amdgcn_fmed3f(y, -60.0f, 60.0f);
  const float e = __builtin_amdgcn_exp2f(c * -1.44269504088896340736f);
  return y * __builtin_amdgcn_rcpf(1.0f + e);
}

__device__ __forceinline__ unsigned chunk_off(int q, int pi) {
  int p, op, i;
  if (q < 16) { p = 0; op = 1; i = q; }
  else if (q >= 112) { p = 3; op = 0; i = q - 112; }
  else { const int j = q - 16; op = j & 1; p = (j >> 5) + op; i = (j & 31) >> 1; }
  return (unsigned)(p * KT_BYTES + op * OP_BYTES + ((i + 2 * pi) & 15) * 32768);
}
__device__ __forceinline__ int chunk_pos(int p, int op, int c, int pi) {
  const int i = (c - 2 * pi) & 15;
  if (op) return p == 0 ? i : 16 + 32 * (p - 1) + 2 * i + 1;
  return p == 3 ? 112 + i : 16 + 32 * p + 2 * i;
}

__global__ __launch_bounds__(512, 2) void fused_kernel(Params P) {
  extern __shared__ __attribute__((aligned(16))) unsigned char smem[];
  LAS unsigned char* lds = (LAS unsigned char*)smem;
  LAS unsigned* ctl = (LAS unsigned*)(lds + CTL_OFF);
  const int tid = threadIdx.x, lane = tid & 63, wid = __builtin_amdgcn_readfirstlane(tid >> 6);
  const int b = blockIdx.x, rank = P.rank;
  char* myws = P.ws_all[rank];
  unsigned* tmo = (unsigned*)(myws + WS_TMO);
  STAMP(0);

  unsigned long long wsp;
  asm volatile("global_load_dwordx2 %0, %1, off" : "=&v"(wsp) : "v"(P.ws_all + (tid & 7)) : "memory");
  unsigned ticket = 0;
  if (tid == 448) asm volatile("global_atomic_add %0, %1, %2, off sc0" : "=&v"(ticket) : "v"(myws + WS_TICKET + (b & 15) * 128), "v"(1u) : "memory");
  const int ntask = b < 28 ? 0 : (b >= 32 && b < 60 ? 2 : 1);
  if (ntask == 0) {
    asm volatile("s_waitcnt vmcnt(0)" : "+v"(ticket), "+v"(wsp) :: "memory");
    if (tid == 448) ctl[4] = ticket / 16 + 1;
    __syncthreads();
  }
  for (int it = 0; it < ntask; ++it) {
    const int task = it == 0 ? b - 28 : 228 + (b - 32);
    const int p = task >> 6, G = task & 63, op = wid < 4 ? 1 : 0, h = wid & 1, j = (wid >> 1) & 1;
    unsigned d[8];
    if (op) {
      const int c = lane & 15, q = lane >> 4;
      unsigned w[16];
      const unsigned vo = (unsigned)(q * 16) * NDIM + c * 4;
      const unsigned char* sbase = P.w + (size_t)(p * 128 + h * 64 + 1) * NDIM + G * 64;
      asm volatile("global_load_dword %0, %16, %17 offset:-4096\n\tglobal_load_dword %1, %16, %17\n\t"
                   "global_load_dword %2, %16, %18 offset:-4096\n\tglobal_load_dword %3, %16, %18\n\t"
                   "global_load_dword %4, %16, %19 offset:-4096\n\tglobal_load_dword %5, %16, %19\n\t"
                   "global_load_dword %6, %16, %20 offset:-4096\n\tglobal_load_dword %7, %16, %20\n\t"
                   "global_load_dword %8, %16, %21 offset:-4096\n\tglobal_load_dword %9, %16, %21\n\t"
                   "global_load_dword %10, %16, %22 offset:-4096\n\tglobal_load_dword %11, %16, %22\n\t"
                   "global_load_dword %12, %16, %23 offset:-4096\n\tglobal_load_dword %13, %16, %23\n\t"
                   "global_load_dword %14, %16, %24 offset:-4096\n\tglobal_load_dword %15, %16, %24\n\t"
                   "s_waitcnt vmcnt(0)"
                   : "=&v"(w[0]), "=&v"(w[1]), "=&v"(w[2]), "=&v"(w[3]), "=&v"(w[4]), "=&v"(w[5]), "=&v"(w[6]), "=&v"(w[7]),
                     "=&v"(w[8]), "=&v"(w[9]), "=&v"(w[10]), "=&v"(w[11]), "=&v"(w[12]), "=&v"(w[13]), "=&v"(w[14]), "=&v"(w[15])
                   : "v"(vo), "s"(sbase), "s"(sbase + 2 * NDIM), "s"(sbase + 4 * NDIM), "s"(sbase + 6 * NDIM), "s"(sbase + 8 * NDIM), "s"(sbase + 10 * NDIM), "s"(sbase + 12 * NDIM), "s"(sbase + 14 * NDIM) : "memory");
      const unsigned sel = j ? 0x07030602u : 0x05010400u;
#pragma unroll
      for (int jj = 0; jj < 4; ++jj) {
        const unsigned tA = __builtin_amdgcn_perm(w[4 * jj + 1], w[4 * jj], sel), tB = __builtin_amdgcn_perm(w[4 * jj + 3], w[4 * jj + 2], sel);
        d[jj] = __builtin_amdgcn_perm(tB, tA, 0x05040100u); d[4 + jj] = __builtin_amdgcn_perm(tB, tA, 0x07060302u);
      }
    } else {
      const int fr = lane & 15, fq = lane >> 4;
      const unsigned char* src = P.x + (size_t)((4 * G + 2 * j) * 16 + fr) * KSH + p * 128 + h * 64 + fq * 16;
      u32x4 a, bq;
      asm volatile("global_load_dwordx4 %0, %2, off\n\tglobal_load_dwordx4 %1, %3, off\n\ts_waitcnt vmcnt(0)" : "=&v"(a), "=&v"(bq) : "v"(src), "v"(src + 16 * KSH) : "memory");
#pragma unroll
      for (int i = 0; i < 4; ++i) { d[i] = a[i]; d[4 + i] = bq[i]; }
    }
    asm volatile("s_waitcnt vmcnt(0)" : "+v"(ticket), "+v"(wsp) :: "memory");
    char* flagp = (char*)wsp + WS_READY + (size_t)(rank * 256 + task) * 4;
    if (tid == 448) ctl[4] = ticket / 16 + 1;
    __syncthreads();
    const unsigned call_ = ctl[4];
    char* sb = myws + WS_SRC + (size_t)(call_ & 1) * SRC_BYTES;
    char* rw = sb + SRC_RAW + (size_t)p * KT_BYTES + op * OP_BYTES + (size_t)(4 * G + 2 * j) * 2048 + h * 1024 + lane * 16;
    st16_sys(rw, (u32x4){d[0], d[1], d[2], d[3]}); st16_sys(rw + 2048, (u32x4){d[4], d[5], d[6], d[7]});
    asm volatile("s_waitcnt vmcnt(0)\n\ts_barrier" ::: "memory");
    if (tid < NR) st4_sys(flagp, call_);
    LAS unsigned char* scr = lds + wid * 4096;
    *(LAS u32x4*)(scr + lane * 16) = (u32x4){d[0], d[1], d[2], d[3]};
    *(LAS u32x4*)(scr + 1024 + lane * 16) = (u32x4){d[4], d[5], d[6], d[7]};
    typedef unsigned short u16x2 __attribute__((ext_vector_type(2)));
    u16x2 mh = {0, 0}, ml = {0, 0};
#pragma unroll
    for (int i = 0; i < 8; ++i) { const unsigned x = d[i] & 0x78787878u;
      mh = __builtin_elementwise_max(mh, __builtin_bit_cast(u16x2, x)); ml = __builtin_elementwise_max(ml, __builtin_bit_cast(u16x2, x << 8)); }
    const u16x2 mm = __builtin_elementwise_max(mh, ml);
    const unsigned lmax = max((unsigned)mm[0] >> 11, (unsigned)mm[1] >> 11);
    int top = 0;
#pragma unroll
    for (int st = 8; st > 0; st >>= 1) if (__builtin_popcountll(__ballot(lmax >= (unsigned)(top + st))) >= 2) top += st;
    const unsigned base = top > 7 ? top - 7 : 0, base8 = base * 0x08080808u;
    unsigned cw[8], M = 0;
#pragma unroll
    for (int i = 0; i < 8; ++i) {
      const unsigned u = ((d[i] & 0x78787878u) | 0x80808080u) - base8;
      const unsigned exc = (~u | (u << 1)) & 0x80808080u;
      cw[i] = (d[i] & 0x87878787u) | (u & 0x38383838u);
      M |= __builtin_amdgcn_udot4(exc >> 7, 0x08040201u, 0u, false) << (4 * i);
    }
#pragma unroll
    for (int i = 0; i < 6; ++i) cw[i] |= (cw[7] << (6 - i)) & 0x40404040u;
    cw[6] |= (cw[7] >> 1) & 0x40404040u;
    unsigned total = 0;
    for (;;) {
      const unsigned long long bal = __ballot(M != 0);
      if (bal == 0 || total > 128) break;
      if (M != 0) {
        const unsigned jb = __builtin_ctz(M); M &= M - 1;
        const unsigned pos = (jb >> 4) * 1024 + lane * 16 + (jb & 15);
        const unsigned e = (scr[pos] >> 3) & 15u, x = e ^ (((e - base) & 7u) + base);
        const unsigned idx = total + __builtin_amdgcn_mbcnt_hi((unsigned)(bal >> 32), __builtin_amdgcn_mbcnt_lo((unsigned)bal, 0u));
        if (idx < 128) *(LAS unsigned short*)(scr + 2048 + idx * 2) = (unsigned short)(((wid & 1) << 11 | pos) << 4 | x);
      }
      total += (unsigned)__builtin_popcountll(bal);
    }
    if (lane == 0) ctl[8 + wid] = total;
    const unsigned ent0 = *(LAS unsigned short*)(scr + 2048 + lane * 2), ent1 = *(LAS unsigned short*)(scr + 2048 + 128 + lane * 2);
    asm volatile("s_waitcnt lgkmcnt(0)\n\ts_barrier" ::: "memory");
    const unsigned t0_ = ctl[8 + (wid & 6)], t1_ = ctl[9 + (wid & 6)];
    const unsigned off = (wid & 1) ? t0_ : 0;
    const bool raw = t0_ + t1_ > 128;
    const unsigned U = ((p * 2 + op) * 64 + G) * 4 + (wid & 3);
    if (!raw) {
      char* pk = sb + SRC_PACK + (size_t)U * 1792;
      st16_sys(pk + lane * 16, (u32x4){cw[0], cw[1], cw[2], cw[3]});
      { typedef unsigned u32x3 __attribute__((ext_vector_type(3))); const u32x3 v3 = {cw[4], cw[5], cw[6]};
        asm volatile("global_store_dwordx3 %0, %1, off sc0 sc1\n\ts_nop 1" :: "v"(pk + 1024 + lane * 12), "v"(v3) : "memory"); }
      char* ex = sb + SRC_EXC + (size_t)(U >> 1) * 256 + (off + lane) * 2;
      if (lane < (int)total) asm volatile("global_store_short %0, %1, off sc0 sc1\n\ts_nop 1" :: "v"(ex), "v"(ent0) : "memory");
      if (lane + 64 < (int)total) asm volatile("global_store_short %0, %1, off offset:128 sc0 sc1\n\ts_nop 1" :: "v"(ex), "v"(ent1) : "memory");
    }
    if (lane == 0) asm volatile("global_store_short %0, %1, off sc0 sc1\n\ts_nop 1" :: "v"(sb + SRC_HDR + U * 2), "v"(raw ? 0x8000u : ((t0_ + t1_) | base << 8)) : "memory");
    asm volatile("s_waitcnt vmcnt(0)\n\ts_barrier" ::: "memory");
    if (tid < NR) st4_sys(flagp + (WS_READY2 - WS_READY), call_);
  }
  const unsigned call = (unsigned)__builtin_amdgcn_readfirstlane(ctl[4]);
  STAMP(1);

  if (b < GEMM0) {
    __syncthreads();
    if (tid < 32) ctl[16 + tid] = 0;
    asm volatile("s_waitcnt lgkmcnt(0)\n\ts_barrier" ::: "memory");
    const unsigned lctl = (unsigned)(size_t)lds + CTL_OFF;
    if (b >= 28) {
      if (b != 28 || wid != 7) return;
      const unsigned pbase = call * 256;
      const int mp = lane >> 3, mo = lane & 7;
      unsigned* pl = (unsigned*)(myws + WS_PROG + mo * 256) + mp;
      const unsigned long long t0 = now();
      {
        const unsigned* fl = (const unsigned*)(myws + WS_READY) + rank * 256 + lane * 4;
        for (;;) {
          u32x4 v; asm volatile("global_load_dwordx4 %0, %1, off sc0 sc1\n\ts_waitcnt vmcnt(0)" : "=&v"(v) : "v"(fl) : "memory");
          if (__all(v[0] >= call && v[1] >= call && v[2] >= call && v[3] >= call)) break;
          if (give_up(t0, tmo, call)) break;
          __builtin_amdgcn_s_sleep(2);
        }
        if (mp == 7) st4_sys(pl, pbase + 128);
      }
      const unsigned* cnt = (const unsigned*)(myws + WS_CNT) + (mp < 7 ? mp : 0) * 128;
      int pos = mp < 7 ? 0 : 128;
      for (;;) {
        const int q = pos + mo < 128 ? pos + mo : 127;
        unsigned v; asm volatile("global_load_dword %0, %1, off sc1\n\ts_waitcnt vmcnt(0)" : "=&v"(v) : "v"(cnt + q) : "memory");
        const unsigned long long m = __ballot(mp < 7 && pos + mo < 128 && v >= call);
        const int adv = __builtin_ctz(~(unsigned)((m >> (mp * 8)) & 0xFF));
        if (adv > 0) { pos += adv; st4_sys(pl, pbase + pos); }
        if (__all(pos >= 128)) { STAMPW(40); break; }
        if (m == 0) { if (give_up(t0, tmo, call)) break; __builtin_amdgcn_s_sleep(MON_SLEEP); }
      }
      return;
    }
    const int pi = b % 7, peer = (rank + 1 + pi) & 7;
    auto lds_rd = [&](unsigned a) -> unsigned { unsigned v; asm volatile("ds_read_b32 %0, %1\n\ts_waitcnt lgkmcnt(0)" : "=v"(v) : "v"(a) : "memory"); return (unsigned)__builtin_amdgcn_readfirstlane(v); };
    const char* srcb = P.ws_all[peer] + WS_SRC + (size_t)(call & 1) * SRC_BYTES;
    const unsigned lhdr = (unsigned)(size_t)lds + 16384, lexc = (unsigned)(size_t)lds + 20480;
    if (wid == 7) {
      unsigned* tk = (unsigned*)(myws + WS_TKT + pi * 128);
      const unsigned base = (call - 1) * 116 - 16;
      const unsigned long long t0 = now();
      const unsigned* rdy = (const unsigned*)(myws + WS_READY) + peer * 256 + lane * 4;
      const unsigned* rdy2 = (const unsigned*)(myws + WS_READY2) + peer * 256 + lane * 4;
      unsigned long long rmask = 0;
      bool hdr_ok = false;
      for (int t = 0;; ++t) {
        if (t >= AHEAD) while (lds_rd(lctl + 4 * (16 + ((t - AHEAD) & 7))) != 8 && now() - t0 < 2 * DEADLINE) __builtin_amdgcn_s_sleep(1);
        unsigned v = (unsigned)(b / 7 + 4 * t);
        if (t >= 4) {
          if (lane == 0) asm volatile("global_atomic_add %0, %1, %2, off sc0 sc1\n\ts_waitcnt vmcnt(0)" : "=&v"(v) : "v"(tk), "v"(1u) : "memory");
          v = (unsigned)__builtin_amdgcn_readfirstlane(v) - base;
        }
        const bool stop = v >= 128 || t >= 254;
        if (!stop && rmask != ~0ull) {
          const unsigned o = chunk_off((int)v, pi);
          const int bit = (o >> 20) * 16 + ((o >> 15) & 15);
          if (!((rmask >> bit) & 1)) {
            const unsigned long long t1 = now();
            for (;;) {
              u32x4 f; asm volatile("global_load_dwordx4 %0, %1, off sc0 sc1\n\ts_waitcnt vmcnt(0)" : "=&v"(f) : "v"(rdy) : "memory");
              rmask = __ballot(f[0] >= call && f[1] >= call && f[2] >= call && f[3] >= call);
              if ((rmask >> bit) & 1) break;
              if (give_up(t1, tmo, call)) { rmask = ~0ull; break; }
              __builtin_amdgcn_s_sleep(2);
            }
          }
        }
        if (lane == 0) asm volatile("ds_write_b32 %0, %1\n\tds_write_b32 %2, %3" :: "v"(lctl + 4 * (16 + (t & 7))), "v"(0u), "v"(lctl + 4 * (32 + (t & 7))),
                                    "v"((unsigned)(t + 1) << 16 | (hdr_ok ? 0u : 256u) | (stop ? 255u : v + 1)) : "memory");
        if (stop) break;
        if (!hdr_ok && t >= 3) {
          u32x4 f; asm volatile("global_load_dwordx4 %0, %1, off sc0 sc1\n\ts_waitcnt vmcnt(0)" : "=&v"(f) : "v"(rdy2) : "memory");
          if (__all(f[0] >= call && f[1] >= call && f[2] >= call && f[3] >= call)) {
            u32x4 h0, h1, h2, h3; const char* hp = srcb + SRC_HDR + lane * 16;
            asm volatile("global_load_dwordx4 %0, %4, off sc0 sc1\n\tglobal_load_dwordx4 %1, %4, off offset:1024 sc0 sc1\n\t"
                         "global_load_dwordx4 %2, %4, off offset:2048 sc0 sc1\n\tglobal_load_dwordx4 %3, %4, off offset:3072 sc0 sc1\n\ts_waitcnt vmcnt(0)"
                         : "=&v"(h0), "=&v"(h1), "=&v"(h2), "=&v"(h3) : "v"(hp) : "memory");
            asm volatile("ds_write_b128 %0, %1\n\tds_write_b128 %0, %2 offset:1024\n\tds_write_b128 %0, %3 offset:2048\n\tds_write_b128 %0, %4 offset:3072\n\ts_waitcnt lgkmcnt(0)"
                         :: "v"(lhdr + lane * 16), "v"(h0), "v"(h1), "v"(h2), "v"(h3) : "memory");
            hdr_ok = true; rmask = ~0ull;
          }
        }
      }
      return;
    }
    char* dstb = myws + WS_DST + (size_t)peer * SHARD_BYTES;
    unsigned* cntb = (unsigned*)(myws + WS_CNT) + pi * 128;
    const unsigned scr = (unsigned)(size_t)lds + wid * 2048;
    struct Pair { int t, q; unsigned hdr, off; };
    auto fetch = [&](int m, Pair& f) -> bool {
      const int t = m >> 3;
      unsigned v = lds_rd(lctl + 4 * (32 + (t & 7)));
      if ((v >> 16) != (unsigned)(t + 1)) {
        const unsigned long long t0 = now();
        while (((v = lds_rd(lctl + 4 * (32 + (t & 7)))) >> 16) != (unsigned)(t + 1)) { if (now() - t0 > 2 * DEADLINE) return false; __builtin_amdgcn_s_sleep(1); }
      }
      if ((v & 255) == 255) return false;
      f.t = t; f.q = (int)(v & 255) - 1;
      const unsigned o = chunk_off(f.q, pi), i = m & 7;
      f.off = o + (i >> 1) * 8192 + (i & 1) * 4096;
      f.hdr = (v & 256) ? 0x80008000u : lds_rd(lhdr + ((o >> 11) + 2 * i) * 2);
      return true;
    };
    const char* locb = myws + WS_SRC + (size_t)(call & 1) * SRC_BYTES;
#define LD2(RA, RB, RB3) "global_load_dwordx4 " RA ", %[l16], %[pa] sc0 sc1\n\t" \
                   "s_cmp_lg_u32 %[raw], 0\n\ts_cbranch_scc1 1f\n\t" \
                   "global_load_dwordx3 " RB3 ", %[l12], %[pa] offset:1024 sc0 sc1\n\ts_branch 2f\n" \
                   "1:\n\tglobal_load_dwordx4 " RB ", %[l16], %[pa] offset:2048 sc0 sc1\n" \
                   "2:"
#define UNIT_INDEX(off_, u_) ((((off_) >> 15) << 4) + ((((off_) >> 13) & 3) << 2) + ((((off_) >> 12) & 1) << 1) + (u_))
#define ISSUE0(f_, valid_) do {     \
      const unsigned h_ = UNI((f_).hdr), o_ = UNI((f_).off), U_ = UNIT_INDEX(o_, 0), raw_ = h_ & 0x8000u, cnt_ = raw_ ? 0u : h_ & 255; \
      const char* sb_ = (valid_) ? srcb : locb; \
      const char* pa_ = sb_ + (raw_ ? SRC_RAW + o_ : SRC_PACK + (size_t)U_ * 1792); \
      const char* ep_ = sb_ + SRC_EXC + (size_t)(U_ >> 1) * 256; const unsigned eo_ = min((unsigned)lane, cnt_ > 1 ? (cnt_ - 1) >> 1 : 0u) * 4; \
      asm volatile(LD2("v[200:203]", "v[204:207]", "v[204:206]") "\n\tglobal_load_dword v208, %[eo], %[pe] sc0 sc1" \
                   : "={v[200:203]}"(a0), "={v[204:207]}"(b0), "={v208}"(ee) \
                   : [l16] "v"(lane * 16), [l12] "v"(lane * 12), [eo] "v"(eo_), [pa] "s"(pa_), [pe] "s"(ep_), [raw] "s"(raw_) : "scc", "memory"); \
    } while (0)
#define ISSUE1(f_, valid_) do {     \
      const unsigned h_ = UNI((f_).hdr) >> 16, o_ = UNI((f_).off), U_ = UNIT_INDEX(o_, 1), raw_ = h_ & 0x8000u; \
      const char* sb_ = (valid_) ? srcb : locb; \
      const char* pa_ = sb_ + (raw_ ? SRC_RAW + o_ + 1024 : SRC_PACK + (size_t)U_ * 1792); \
      asm volatile(LD2("v[210:213]", "v[214:217]", "v[214:216]") \
                   : "={v[210:213]}"(a1), "={v[214:217]}"(b1) \
                   : [l16] "v"(lane * 16), [l12] "v"(lane * 12), [pa] "s"(pa_), [raw] "s"(raw_) : "scc", "memory"); \
    } while (0)
    auto arrive = [&](const Pair& f) {
      unsigned old = 0;
      if (lane == 0) asm volatile("ds_add_rtn_u32 %0, %1, %2\n\ts_waitcnt lgkmcnt(0)" : "=&v"(old) : "v"(lctl + 4 * (16 + (f.t & 7))), "v"(1u) : "memory");
      if (__builtin_amdgcn_readfirstlane(old) == 7 && lane == 0) st4_sys(cntb + f.q, call);
    };
    auto process = [&](u32x4 a, u32x4 bq, unsigned e, const Pair& f, int u) {
      u32x4 oa = a, ob = bq;
      const unsigned h0 = UNI(f.hdr), fh = u ? h0 >> 16 : h0;
      if (!(fh & 0x8000u)) {
        const unsigned base8 = ((fh >> 8) & 15) * 0x08080808u, cnt = h0 & 255;
        unsigned c7 = (bq[2] << 1) & 0x80808080u;
        c7 |= (a[0] >> 6) & 0x01010101u; c7 |= (a[1] >> 5) & 0x02020202u; c7 |= (a[2] >> 4) & 0x04040404u; c7 |= (a[3] >> 3) & 0x08080808u;
        c7 |= (bq[0] >> 2) & 0x10101010u; c7 |= (bq[1] >> 1) & 0x20202020u;
#pragma unroll
        for (int i = 0; i < 4; ++i) oa[i] = (a[i] & 0xBFBFBFBFu) + base8;
#pragma unroll
        for (int i = 0; i < 3; ++i) ob[i] = (bq[i] & 0xBFBFBFBFu) + base8;
        ob[3] = c7 + base8;
        if (cnt > 0) {
          const unsigned e0 = e & 0xFFFFu, e1 = e >> 16;
          asm volatile("ds_write_b128 %2, %0\n\tds_write_b128 %2, %1 offset:1024" :: "v"(oa), "v"(ob), "v"(scr + lane * 16) : "memory");
          if (2u * lane < cnt && (int)(e0 >> 15) == u) asm volatile("ds_xor_b32 %0, %1" :: "v"(scr + ((e0 >> 4) & 0x7FCu)), "v"((e0 & 15u) << (3 + 8 * ((e0 >> 4) & 3))) : "memory");
          if (2u * lane + 1 < cnt && (int)(e1 >> 15) == u) asm volatile("ds_xor_b32 %0, %1" :: "v"(scr + ((e1 >> 4) & 0x7FCu)), "v"((e1 & 15u) << (3 + 8 * ((e1 >> 4) & 3))) : "memory");
          asm volatile("ds_read_b128 %0, %2\n\tds_read_b128 %1, %2 offset:1024\n\ts_waitcnt lgkmcnt(0)" : "=&v"(oa), "=&v"(ob) : "v"(scr + lane * 16) : "memory");
        }
      }
      char* dp = dstb + UNI(f.off) + u * 1024;
      asm volatile("global_store_dwordx4 %0, %1, %3 sc0 sc1\n\tglobal_store_dwordx4 %0, %2, %3 offset:2048 sc0 sc1\n\ts_nop 1" :: "v"(lane * 16), "v"(oa), "v"(ob), "s"(dp) : "memory");
    };
    u32x4 a0, b0, a1, b1; unsigned ee;
    Pair f = {0, 0, 0x80008000u, 0u}, fn = f, prev = f;
    bool v = fetch(wid, f), have_prev = false;
    STAMPW(20 + wid);
    if (v) {
      ISSUE0(f, true); ISSUE1(f, true);
      asm volatile("s_waitcnt vmcnt(2)" : "+{v[200:203]}"(a0), "+{v[204:207]}"(b0), "+{v208}"(ee) :: "memory");
      int nstamp = 0;
      for (int m = wid;; m += 7) {
        const unsigned e = ee;
        unsigned ecur = e; asm volatile("" : "+v"(ecur));
        process(a0, b0, ecur, f, 0);
        const bool vn = fetch(m + 7, fn);
        ISSUE0(fn, vn);
        asm volatile("s_waitcnt vmcnt(5)" : "+{v[210:213]}"(a1), "+{v[214:217]}"(b1) :: "memory");
        if (have_prev) arrive(prev);
        process(a1, b1, ecur, f, 1);
        ISSUE1(fn, vn);
        asm volatile("s_waitcnt vmcnt(4)" : "+{v[200:203]}"(a0), "+{v[204:207]}"(b0), "+{v208}"(ee) :: "memory");
        prev = f; have_prev = true;
        if (!vn) break;
        f = fn;
      }
      asm volatile("s_waitcnt vmcnt(0)" ::: "memory");
      arrive(prev);
    }
#undef ISSUE0
#undef ISSUE1
#undef LD2
#undef UNIT_INDEX
    STAMPW(10 + wid);
    return;
  }

  const int g = b - GEMM0, xcd = g & 7, idx = g >> 3;
  const int pm = (xcd & 1) * 7 + idx % 7, pn = (xcd >> 1) * 4 + idx / 7;
  const int g0 = tile_start(pm), g1 = tile_start(pm + 1), wr = wid >> 2, wc = wid & 3;
  const bool a4 = wid < (g1 - g0) * 2 - 32;
  const char* own = myws + WS_SRC + (size_t)(call & 1) * SRC_BYTES + SRC_RAW;
  const char* dstb = myws + WS_DST;
  const size_t aoff = (size_t)g0 * 2048, boff = OP_BYTES + (size_t)pn * B_BYTES;
  auto kt_base = [&](int kt) -> const char* { const int s = kt >> 2, p = kt & 3; return (s == rank ? own : dstb + (size_t)s * SHARD_BYTES) + (size_t)p * KT_BYTES; };

  unsigned pollA, thr;
  {
    const int kt = lane & 31, s = kt >> 2, p = kt & 3, pi = (s - rank - 1) & 7;
    const int qa = chunk_pos(p, 0, g0 >> 4, pi), qb = chunk_pos(p, 0, (g1 - 1) >> 4, pi), qc = chunk_pos(p, 1, pn, pi);
    pollA = (unsigned)WS_PROG + (unsigned)(g & (NREP - 1)) * 256 + pi * 4;
    thr = call * 256 + (pi == 7 ? 128 : max(max(qa, qb), qc) + 1);
  }

  f32x4 acc[10][4];
#pragma unroll
  for (int m = 0; m < 10; ++m)
#pragma unroll
    for (int n = 0; n < 4; ++n) acc[m][n] = (f32x4){0.f, 0.f, 0.f, 0.f};

  const unsigned lbase = (unsigned)(size_t)lds;
  const unsigned voff = wid * 1024 + lane * 16;
  unsigned pa = lbase + wr * (10 * 2048) + lane * 16, pb = lbase + A_BYTES + wc * (4 * 2048) + lane * 16;
  unsigned npa = pa + BUF_BYTES, npb = pb + BUF_BYTES;
  const unsigned ctl_addr = lbase + CTL_OFF;
  unsigned ready = 0, taken = 0;
  auto wait_tile = [&]() {
    if (wid == 0) {
      const unsigned long long t0 = now();
      for (;;) {
        unsigned q; asm volatile("global_load_dword %0, %1, %2 sc1\n\ts_waitcnt vmcnt(0)" : "=&v"(q) : "v"(pollA), "s"(myws) : "memory");
        unsigned m = (unsigned)__ballot(q >= thr);
        if ((m & ~taken) == 0 && give_up(t0, tmo, call)) m = ~0u;
        if ((m & ~taken) != 0) { if (lane == 0) asm volatile("ds_write_b32 %0, %1" :: "v"(ctl_addr + 8), "v"(m) : "memory"); break; }
        __builtin_amdgcn_s_sleep(POLL_SLEEP);
      }
    }
    unsigned mr;
    asm volatile("s_waitcnt lgkmcnt(0)\n\ts_barrier\n\tds_read_b32 %0, %1\n\ts_waitcnt lgkmcnt(0)" : "=v"(mr) : "v"(ctl_addr + 8) : "memory");
    ready = (unsigned)__builtin_amdgcn_readfirstlane(mr);
  };
  wait_tile();
  STAMP(2);
  Regs r;
  { const int k = __builtin_ctz(ready); taken |= 1u << k; const char* kb = kt_base(k);
    stage_all(kb + aoff, kb + boff, voff, lbase + wid * 1024, a4);
    asm volatile("s_waitcnt vmcnt(0)\n\ts_barrier" ::: "memory");
    prime_reads(r, pa, pb); }
  int cur = 0;
  for (int n = 0; n < 32; ++n) {
    const unsigned nl = lbase + (cur ^ 1) * BUF_BYTES + wid * 1024;
    const unsigned la = nl, lb = nl + A_BYTES;
    unsigned q0;
    const unsigned ctl_cur = ctl_addr + (n & 1) * 4;
#define DB(i) dma(gb + (i) * 8192, voff, lb + (i) * 8192)
#define DA(i) dma(ga + (i) * 8192, voff, la + (i) * 8192)
    auto try_stage = [&](int path) -> bool {
      const unsigned cand = ready & ~taken;
      if (cand == 0) return false;
      const int k = __builtin_ctz(cand); taken |= 1u << k;
      if (wr == 1) {
        const char* kb = kt_base(k);
        const char* ga = kb + aoff; const char* gb = kb + boff;
        const unsigned vo2 = voff - 4096, la2 = la - 4096, lb2 = lb - 4096;
        dma(gb, vo2, lb2); DB(0); dma(gb + 8192, vo2, lb2 + 8192); DB(1); dma(gb + 2 * 8192, vo2, lb2 + 2 * 8192); DB(2); dma(gb + 3 * 8192, vo2, lb2 + 3 * 8192); DB(3);
        dma(ga, vo2, la2); DA(0); dma(ga + 8192, vo2, la2 + 8192); DA(1); dma(ga + 2 * 8192, vo2, la2 + 2 * 8192); DA(2); dma(ga + 3 * 8192, vo2, la2 + 3 * 8192); DA(3);
        dma(ga + 4 * 8192, vo2, la2 + 4 * 8192); if (a4) DA(4);
      }
      return true;
    };
#define POLL_ISSUE() do { if (wid == 0) asm volatile("global_load_dword %0, %1, %2 sc1" : "=&v"(q0) : "v"(pollA), "s"(myws) : "memory"); } while (0)
#define POLL_PUBLISH(addr) do { if (wid == 0) {   \
      asm volatile("s_waitcnt vmcnt(0)" : "+v"(q0) :: "memory"); \
      const unsigned m_ = (unsigned)__ballot(q0 >= thr); \
      if (lane == 0) asm volatile("ds_write_b32 %0, %1" :: "v"(addr), "v"(m_) : "memory"); } } while (0)
    POLL_ISSUE();
    bool stage = try_stage(0);
#define CHECKPOINT(word, path) do { if (!stage && n < 31) { \
      POLL_PUBLISH(ctl_addr + (word)); \
      unsigned mm; \
      asm volatile("s_waitcnt lgkmcnt(0)\n\ts_barrier\n\tds_read_b32 %0, %1\n\ts_waitcnt lgkmcnt(0)" : "=v"(mm) : "v"(ctl_addr + (word)) : "memory"); \
      ready = (unsigned)__builtin_amdgcn_readfirstlane(mm); \
      POLL_ISSUE(); \
      stage = try_stage(path); } } while (0)
    __builtin_amdgcn_s_setprio(1);
    WL(10); MM(acc[0][0], R_B0, r.b0, R_AX, r.ax);
    WL(8); MM(acc[0][1], R_B1, r.b1, R_AX, r.ax);
    WL(6); MM(acc[0][2], R_B2, r.b2, R_AX, r.ax);
    WL(4); MM(acc[0][3], R_B3, r.b3, R_AX, r.ax);
    RD_AX(pa, 3 * 2048);
    WL(4); MMA4Y(1); RD_AY(pa, 4 * 2048);
    WL(4); MMA4Z(2); RD_AZ(pa, 5 * 2048);
    CHECKPOINT(12, 1);
    WL(4); MMA4X(3); RD_AX(pa, 6 * 2048);
    WL(4); MMA4Y(4); RD_AY(pa, 7 * 2048);
    WL(4); MMA4Z(5); RD_AZ(pa, 8 * 2048);
    if (stage || n == 31) POLL_ISSUE();
    CHECKPOINT(16, 3);
    WL(4); MMA4X(6);
    WL(2); MMA4Y(7); RD_AY(pa, 9 * 2048);
    WL(2); MMA4Z(8);
    POLL_PUBLISH(ctl_cur);
#undef CHECKPOINT
    asm volatile("s_waitcnt vmcnt(0) lgkmcnt(0)\n\ts_barrier" ::: "memory");
    unsigned mreg;
    asm volatile("ds_read_b32 %0, %1" : "=v"(mreg) : "v"(ctl_cur));
    RD_AX(npa, 0);
    asm volatile(
      "s_cmp_lg_u32 %[wr], 0\n\t"
      "s_cbranch_scc1 1f\n\t"
      "v_mfma_f32_16x16x128_f8f6f4 %[c0], v[200:207], v[240:247], %[c0]\n\t"
      "ds_read_b128 v[200:203], %[pb] offset:0\n\tds_read_b128 v[204:207], %[pb] offset:1024\n\t"
      "v_mfma_f32_16x16x128_f8f6f4 %[c1], v[208:215], v[240:247], %[c1]\n\t"
      "ds_read_b128 v[208:211], %[pb] offset:2048\n\tds_read_b128 v[212:215], %[pb] offset:3072\n\t"
      "v_mfma_f32_16x16x128_f8f6f4 %[c2], v[216:223], v[240:247], %[c2]\n\t"
      "ds_read_b128 v[216:219], %[pb] offset:4096\n\tds_read_b128 v[220:223], %[pb] offset:5120\n\t"
      "v_mfma_f32_16x16x128_f8f6f4 %[c3], v[224:231], v[240:247], %[c3]\n\t"
      "ds_read_b128 v[224:227], %[pb] offset:6144\n\tds_read_b128 v[228:231], %[pb] offset:7168\n\t"
      "s_branch 2f\n"
      "1:\n\t"
      "ds_read_b128 v[200:203], %[pb] offset:0\n\tds_read_b128 v[204:207], %[pb] offset:1024\n\t"
      "ds_read_b128 v[208:211], %[pb] offset:2048\n\tds_read_b128 v[212:215], %[pb] offset:3072\n\t"
      "ds_read_b128 v[216:219], %[pb] offset:4096\n\tds_read_b128 v[220:223], %[pb] offset:5120\n\t"
      "ds_read_b128 v[224:227], %[pb] offset:6144\n\tds_read_b128 v[228:231], %[pb] offset:7168\n"
      "2:"
      : [c0] "+v"(acc[9][0]), [c1] "+v"(acc[9][1]), [c2] "+v"(acc[9][2]), [c3] "+v"(acc[9][3]),
        "+{v[200:207]}"(r.b0), "+{v[208:215]}"(r.b1), "+{v[216:223]}"(r.b2), "+{v[224:231]}"(r.b3)
      : [wr] "s"(wr), [pb] "v"(npb), "{v[240:247]}"(r.ay) : "scc");
    RD_AY(npa, 2048); RD_AZ(npa, 2 * 2048);
    __builtin_amdgcn_s_setprio(0);
    asm volatile("s_waitcnt lgkmcnt(14)" : "+v"(mreg));
    ready = (unsigned)__builtin_amdgcn_readfirstlane(mreg);
#undef DA
#undef DB
#undef POLL_ISSUE
#undef POLL_PUBLISH
    if (n == 31) break;
    if (!stage) {
      if ((ready & ~taken) == 0) wait_tile();
      const int k2 = __builtin_ctz(ready & ~taken); taken |= 1u << k2;
      const char* kb2 = kt_base(k2);
      stage_all(kb2 + aoff, kb2 + boff, voff, nl, a4);
      asm volatile("s_waitcnt vmcnt(0)\n\ts_barrier" ::: "memory");
      prime_reads(r, npa, npb);
    }
    { unsigned t = pa; pa = npa; npa = t; t = pb; pb = npb; npb = t; cur ^= 1; }
  }
  asm volatile("s_waitcnt lgkmcnt(0)\n\ts_nop 15\n\ts_nop 15\n\ts_barrier" ::: "memory");
  STAMP(3);

  {
    float s = P.sx[0] * P.sw[0];
    const bool bad = (unsigned)__builtin_amdgcn_readfirstlane(ld4_sys(tmo)) == call;
    if (bad) s = __builtin_nanf("");
    LAS unsigned char* reg = lds + wid * (2 * 16 * 272);
    const int fr = lane & 15, fq = lane >> 4;
#pragma unroll
    for (int m = 0; m < 10; ++m) {
      const int gg = g0 + wr * 10 + m;
      if (gg < g1 && (wr == 0 || m < 9)) {
        LAS unsigned char* rr = reg + (m & 1) * (16 * 272);
#pragma unroll
        for (int j = 0; j < 4; ++j) {
          f32x4 v = {silu(acc[m][0][j], s), silu(acc[m][1][j], s), silu(acc[m][2][j], s), silu(acc[m][3][j], s)};
          *(LAS f32x4*)(rr + fr * 272 + fq * 64 + j * 16) = v;
        }
        float* o = P.out + (size_t)(gg * 16 + fq) * NDIM + pn * 256 + wc * 64 + fr * 4;
#pragma unroll
        for (int r4 = 0; r4 < 4; ++r4) {
          f32x4 v = *(const LAS f32x4*)(rr + (r4 * 4 + fq) * 272 + fr * 16);
          *(f32x4*)(o + (size_t)r4 * 4 * NDIM) = v;
        }
      }
    }
  }
}

}

extern "C" void kernel_launch(int rank, int nranks, ncclComm_t comm, hipStream_t stream,
                              void* const* d_in, const int* in_sizes, int n_in,
                              void* d_out, int out_size, const int* in_dtypes, int out_dtype,
                              void* const* d_out_all, void* const* d_ws_all, size_t ws_bytes) {
  (void)comm; (void)in_sizes; (void)n_in; (void)out_size; (void)in_dtypes; (void)out_dtype; (void)d_out_all;
  if (nranks != NR || (unsigned)rank >= (unsigned)NR || ws_bytes < WS_END) { fprintf(stderr, "unsupported configuration\n"); return; }
  static bool attr_done[NR] = {};
  if (!attr_done[rank]) {
    HIP_CHECK(hipFuncSetAttribute((const void*)fused_kernel, hipFuncAttributeMaxDynamicSharedMemorySize, LDS_BYTES));
    attr_done[rank] = true;
  }
  Params p;
  p.x = (const unsigned char*)d_in[0]; p.w = (const unsigned char*)d_in[1]; p.sx = (const float*)d_in[2]; p.sw = (const float*)d_in[3];
  p.out = (float*)d_out; p.ws_all = (char* const*)d_ws_all; p.rank = rank; p.pad = 0;
  hipLaunchKernelGGL(fused_kernel, dim3(NWG), dim3(512), LDS_BYTES, stream, p);
  HIP_CHECK(hipGetLastError());
}
```

```cpp
#include <hip/hip_runtime.h>
#include <rccl/rccl.h>
#include <cstdio>

#define HIP_CHECK(expr) do { hipError_t _e = (expr); if (_e != hipSuccess) { \
    fprintf(stderr, "HIP error %s at %s:%d\n", hipGetErrorString(_e), __FILE__, __LINE__); \
    return; } } while (0)

namespace {

typedef unsigned u32x4 __attribute__((ext_vector_type(4)));
typedef int i32x8 __attribute__((ext_vector_type(8)));
typedef float f32x4 __attribute__((ext_vector_type(4)));
#define LAS __attribute__((address_space(3)))

constexpr int NR = 8;
constexpr int NDIM = 4096, KSH = 512;
constexpr int NPULL = 28;
constexpr int GEMM0 = 32;
constexpr int NREP = 8;
constexpr int NWG = 256;
constexpr int NTM = 14;
constexpr int A_BYTES = 19 * 2048, B_BYTES = 16 * 2048, BUF_BYTES = A_BYTES + B_BYTES;
constexpr int CTL_OFF = 2 * BUF_BYTES;
constexpr int LDS_BYTES = CTL_OFF + 2048;
constexpr int LEAD = 2;
constexpr int NSTATIC = 4;
constexpr int AHEAD = 4;
constexpr size_t OP_BYTES = 256 * 2048;
constexpr size_t KT_BYTES = 2 * OP_BYTES;
constexpr size_t SHARD_BYTES = 4 * KT_BYTES;
constexpr size_t SRC_RAW = 0, SRC_PACK = SHARD_BYTES, SRC_EXC = SRC_PACK + 2048 * 1792, SRC_HDR = SRC_EXC + 1024 * 256, SRC_BYTES = 2 * SHARD_BYTES;
static_assert(SRC_HDR + 4096 <= SRC_BYTES, "SRC layout");
constexpr size_t WS_SRC = 0;
constexpr size_t WS_DST = 2 * SRC_BYTES;
constexpr size_t WS_READY = WS_DST + NR * SHARD_BYTES;
constexpr size_t WS_READY2 = WS_READY + NR * 256 * 4;
constexpr size_t WS_CNT = WS_READY2 + NR * 256 * 4;
constexpr size_t WS_TKT = WS_CNT + 4096;
constexpr size_t WS_PROG = WS_TKT + 1024;
constexpr size_t WS_TICKET = WS_PROG + NREP * 256;
constexpr size_t WS_TMO = WS_TICKET + 16 * 128;
constexpr size_t WS_END = WS_TMO + 128;
constexpr unsigned long long DEADLINE = 20000000ull;

struct Params {
  const unsigned char* x; const unsigned char* w; const float* sx; const float* sw; float* out;
  char* const* ws_all; int rank; int pad;
};

#define SB() __builtin_amdgcn_sched_barrier(0)
#define UNI(x) ((unsigned)__builtin_amdgcn_readfirstlane((int)(x)))
__device__ __forceinline__ unsigned long long now() { return __builtin_amdgcn_s_memrealtime(); }
__device__ __forceinline__ void st16_sys(void* p, u32x4 v) { asm volatile("global_store_dwordx4 %0, %1, off sc0 sc1\n\ts_nop 1" :: "v"(p), "v"(v) : "memory"); }
__device__ __forceinline__ void st4_sys(void* p, unsigned v) { asm volatile("global_store_dword %0, %1, off sc0 sc1\n\ts_nop 1" :: "v"(p), "v"(v) : "memory"); }
__device__ __forceinline__ unsigned ld4_sys(const void* p) { unsigned v; asm volatile("global_load_dword %0, %1, off sc0 sc1\n\ts_waitcnt vmcnt(0)" : "=&v"(v) : "v"(p) : "memory"); return v; }
__device__ __forceinline__ int tile_start(int pm) { return (pm * 256) / NTM; }

__device__ __forceinline__ bool give_up(unsigned long long t0, unsigned* tmo, unsigned call) {
  if ((unsigned)__builtin_amdgcn_readfirstlane(ld4_sys(tmo)) == call) return true;
  if (now() - t0 > DEADLINE) { st4_sys(tmo, call); return true; }
  return false;
}


#define R_B0 "v[200:207]"
#define R_B1 "v[208:215]"
#define R_B2 "v[216:223]"
#define R_B3 "v[224:231]"
#define R_AX "v[232:239]"
#define R_AY "v[240:247]"
#define R_AZ "v[248:255]"
#define RD_(f, LO, HI, ALL, addr, OFF) asm volatile("ds_read_b128 " LO ", %1 offset:%c2\n\tds_read_b128 " HI ", %1 offset:%c3" : "={" ALL "}"(f) : "v"(addr), "i"(OFF), "i"((OFF) + 1024))
#define RD_B0(addr, OFF) RD_(r.b0, "v[200:203]", "v[204:207]", R_B0, addr, OFF)
#define RD_B1(addr, OFF) RD_(r.b1, "v[208:211]", "v[212:215]", R_B1, addr, OFF)
#define RD_B2(addr, OFF) RD_(r.b2, "v[216:219]", "v[220:223]", R_B2, addr, OFF)
#define RD_B3(addr, OFF) RD_(r.b3, "v[224:227]", "v[228:231]", R_B3, addr, OFF)
#define RD_AX(addr, OFF) RD_(r.ax, "v[232:235]", "v[236:239]", R_AX, addr, OFF)
#define RD_AY(addr, OFF) RD_(r.ay, "v[240:243]", "v[244:247]", R_AY, addr, OFF)
#define RD_AZ(addr, OFF) RD_(r.az, "v[248:251]", "v[252:255]", R_AZ, addr, OFF)
#define WL(N) asm volatile("s_waitcnt lgkmcnt(" #N ")")
#define MM(ACC, RB, vb, RA, va) asm volatile("v_mfma_f32_16x16x128_f8f6f4 %0, " RA ", " RB ", %0" : "+v"(ACC) : "{" RB "}"(vb), "{" RA "}"(va))
#define MMA4X(m) do { MM(acc[m][0], R_B0, r.b0, R_AX, r.ax); MM(acc[m][1], R_B1, r.b1, R_AX, r.ax); MM(acc[m][2], R_B2, r.b2, R_AX, r.ax); MM(acc[m][3], R_B3, r.b3, R_AX, r.ax); } while (0)
#define MMA4Y(m) do { MM(acc[m][0], R_B0, r.b0, R_AY, r.ay); MM(acc[m][1], R_B1, r.b1, R_AY, r.ay); MM(acc[m][2], R_B2, r.b2, R_AY, r.ay); MM(acc[m][3], R_B3, r.b3, R_AY, r.ay); } while (0)
#define MMA4Z(m) do { MM(acc[m][0], R_B0, r.b0, R_AZ, r.az); MM(acc[m][1], R_B1, r.b1, R_AZ, r.az); MM(acc[m][2], R_B2, r.b2, R_AZ, r.az); MM(acc[m][3], R_B3, r.b3, R_AZ, r.az); } while (0)
struct Regs { i32x8 b0, b1, b2, b3, ax, ay, az; };
__device__ __forceinline__ void prime_reads(Regs& r, unsigned pa, unsigned pb) {
  RD_AX(pa, 0); RD_B0(pb, 0); RD_B1(pb, 2048); RD_B2(pb, 4096); RD_B3(pb, 6144); RD_AY(pa, 2048); RD_AZ(pa, 2 * 2048);
}
__device__ __forceinline__ void dma(const char* sbase, unsigned voff, unsigned ldsa) {
  asm volatile("s_mov_b32 m0, %2\n\ts_nop 0\n\tglobal_load_lds_dwordx4 %0, %1 sc0 sc1" :: "v"(voff), "s"(sbase), "s"(ldsa) : "memory");
}
__device__ __forceinline__ void stage_all(const char* ga, const char* gb, unsigned voff, unsigned nl, bool a4) {
  const unsigned la = nl, lb = nl + A_BYTES;
  dma(gb, voff, lb); dma(gb + 8192, voff, lb + 8192); dma(gb + 2 * 8192, voff, lb + 2 * 8192); dma(gb + 3 * 8192, voff, lb + 3 * 8192);
  dma(ga, voff, la); dma(ga + 8192, voff, la + 8192); dma(ga + 2 * 8192, voff, la + 2 * 8192); dma(ga + 3 * 8192, voff, la + 3 * 8192);
  if (a4) dma(ga + 4 * 8192, voff, la + 4 * 8192);
}
__device__ __forceinline__ unsigned chunk_off(int q, int pi) {
  int op, i;
  if (q < LEAD) { op = 1; i = q; }
  else if (q >= 128 - LEAD) { op = 0; i = q - 64; }
  else { const int j = q - LEAD; op = j & 1; i = (j >> 1) + (op ? LEAD : 0); }
  const int p = i >> 4;
  return (unsigned)(p * KT_BYTES + op * OP_BYTES + (((i & 15) + 2 * pi) & 15) * 32768);
}
__device__ __forceinline__ int chunk_pos(int p, int op, int c, int pi) {
  const int i = 16 * p + ((c - 2 * pi) & 15);
  if (op) return i < LEAD ? i : LEAD + 2 * (i - LEAD) + 1;
  return i >= 64 - LEAD ? i + 64 : LEAD + 2 * i;
}

__global__ __launch_bounds__(512, 2) void fused_kernel(Params P) {
  extern __shared__ __attribute__((aligned(16))) unsigned char smem[];
  LAS unsigned char* lds = (LAS unsigned char*)smem;
  LAS unsigned* ctl = (LAS unsigned*)(lds + CTL_OFF);
  const int tid = threadIdx.x, lane = tid & 63, wid = __builtin_amdgcn_readfirstlane(tid >> 6);
  const int b = blockIdx.x, rank = P.rank;
  char* myws = P.ws_all[rank];
  unsigned* tmo = (unsigned*)(myws + WS_TMO);

  unsigned long long wsp;
  asm volatile("global_load_dwordx2 %0, %1, off" : "=&v"(wsp) : "v"(P.ws_all + (tid & 7)) : "memory");
  unsigned ticket = 0;
  if (tid == 448) asm volatile("global_atomic_add %0, %1, %2, off sc0" : "=&v"(ticket) : "v"(myws + WS_TICKET + (b & 15) * 128), "v"(1u) : "memory");
  const int ntask = b < NPULL ? 0 : (b >= GEMM0 && b < GEMM0 + NPULL ? 2 : 1);
  if (ntask == 0) {
    asm volatile("s_waitcnt vmcnt(0)" : "+v"(ticket), "+v"(wsp) :: "memory");
    if (tid == 448) ctl[4] = ticket / 16 + 1;
    __syncthreads();
  }
  for (int it = 0; it < ntask; ++it) {
    const int task = it == 0 ? b - NPULL : NWG - NPULL + (b - GEMM0);
    const int p = task >> 6, G = task & 63, op = wid < 4 ? 1 : 0, h = wid & 1, j = (wid >> 1) & 1;
    unsigned d[8];
    if (op) {
      const int c = lane & 15, q = lane >> 4;
      unsigned w[16];
      const unsigned vo = (unsigned)(q * 16) * NDIM + c * 4;
      const unsigned char* sbase = P.w + (size_t)(p * 128 + h * 64 + 1) * NDIM + G * 64;
      asm volatile("global_load_dword %0, %16, %17 offset:-4096\n\tglobal_load_dword %1, %16, %17\n\t"
                   "global_load_dword %2, %16, %18 offset:-4096\n\tglobal_load_dword %3, %16, %18\n\t"
                   "global_load_dword %4, %16, %19 offset:-4096\n\tglobal_load_dword %5, %16, %19\n\t"
                   "global_load_dword %6, %16, %20 offset:-4096\n\tglobal_load_dword %7, %16, %20\n\t"
                   "global_load_dword %8, %16, %21 offset:-4096\n\tglobal_load_dword %9, %16, %21\n\t"
                   "global_load_dword %10, %16, %22 offset:-4096\n\tglobal_load_dword %11, %16, %22\n\t"
                   "global_load_dword %12, %16, %23 offset:-4096\n\tglobal_load_dword %13, %16, %23\n\t"
                   "global_load_dword %14, %16, %24 offset:-4096\n\tglobal_load_dword %15, %16, %24\n\t"
                   "s_waitcnt vmcnt(0)"
                   : "=&v"(w[0]), "=&v"(w[1]), "=&v"(w[2]), "=&v"(w[3]), "=&v"(w[4]), "=&v"(w[5]), "=&v"(w[6]), "=&v"(w[7]),
                     "=&v"(w[8]), "=&v"(w[9]), "=&v"(w[10]), "=&v"(w[11]), "=&v"(w[12]), "=&v"(w[13]), "=&v"(w[14]), "=&v"(w[15])
                   : "v"(vo), "s"(sbase), "s"(sbase + 2 * NDIM), "s"(sbase + 4 * NDIM), "s"(sbase + 6 * NDIM), "s"(sbase + 8 * NDIM), "s"(sbase + 10 * NDIM), "s"(sbase + 12 * NDIM), "s"(sbase + 14 * NDIM) : "memory");
      const unsigned sel = j ? 0x07030602u : 0x05010400u;
#pragma unroll
      for (int jj = 0; jj < 4; ++jj) {
        const unsigned tA = __builtin_amdgcn_perm(w[4 * jj + 1], w[4 * jj], sel), tB = __builtin_amdgcn_perm(w[4 * jj + 3], w[4 * jj + 2], sel);
        d[jj] = __builtin_amdgcn_perm(tB, tA, 0x05040100u); d[4 + jj] = __builtin_amdgcn_perm(tB, tA, 0x07060302u);
      }
    } else {
      const int fr = lane & 15, fq = lane >> 4;
      const unsigned char* src = P.x + (size_t)((4 * G + 2 * j) * 16 + fr) * KSH + p * 128 + h * 64 + fq * 16;
      u32x4 a, bq;
      asm volatile("global_load_dwordx4 %0, %2, off\n\tglobal_load_dwordx4 %1, %3, off\n\ts_waitcnt vmcnt(0)" : "=&v"(a), "=&v"(bq) : "v"(src), "v"(src + 16 * KSH) : "memory");
#pragma unroll
      for (int i = 0; i < 4; ++i) { d[i] = a[i]; d[4 + i] = bq[i]; }
    }
    asm volatile("s_waitcnt vmcnt(0)" : "+v"(ticket), "+v"(wsp) :: "memory");
    char* flagp = (char*)wsp + WS_READY + (size_t)(rank * 256 + task) * 4;
    if (tid == 448) ctl[4] = ticket / 16 + 1;
    __syncthreads();
    const unsigned call_ = ctl[4];
    char* sb = myws + WS_SRC + (size_t)(call_ & 1) * SRC_BYTES;
    char* rw = sb + SRC_RAW + (size_t)p * KT_BYTES + op * OP_BYTES + (size_t)(4 * G + 2 * j) * 2048 + h * 1024 + lane * 16;
    st16_sys(rw, (u32x4){d[0], d[1], d[2], d[3]}); st16_sys(rw + 2048, (u32x4){d[4], d[5], d[6], d[7]});
    asm volatile("s_waitcnt vmcnt(0)\n\ts_barrier" ::: "memory");
    if (tid < NR) st4_sys(flagp, call_);
    LAS unsigned char* scr = lds + wid * 4096;
    *(LAS u32x4*)(scr + lane * 16) = (u32x4){d[0], d[1], d[2], d[3]};
    *(LAS u32x4*)(scr + 1024 + lane * 16) = (u32x4){d[4], d[5], d[6], d[7]};
    typedef unsigned short u16x2 __attribute__((ext_vector_type(2)));
    u16x2 mh = {0, 0}, ml = {0, 0};
#pragma unroll
    for (int i = 0; i < 8; ++i) { const unsigned x = d[i] & 0x78787878u;
      mh = __builtin_elementwise_max(mh, __builtin_bit_cast(u16x2, x)); ml = __builtin_elementwise_max(ml, __builtin_bit_cast(u16x2, x << 8)); }
    const u16x2 mm = __builtin_elementwise_max(mh, ml);
    const unsigned lmax = max((unsigned)mm[0] >> 11, (unsigned)mm[1] >> 11);
    int top = 0;
#pragma unroll
    for (int st = 8; st > 0; st >>= 1) if (__builtin_popcountll(__ballot(lmax >= (unsigned)(top + st))) >= 2) top += st;
    const unsigned base = top > 7 ? top - 7 : 0, base8 = base * 0x08080808u;
    unsigned cw[8], M = 0;
#pragma unroll
    for (int i = 0; i < 8; ++i) {
      const unsigned u = ((d[i] & 0x78787878u) | 0x80808080u) - base8;
      const unsigned exc = (~u | (u << 1)) & 0x80808080u;
      cw[i] = (d[i] & 0x87878787u) | (u & 0x38383838u);
      M |= __builtin_amdgcn_udot4(exc >> 7, 0x08040201u, 0u, false) << (4 * i);
    }
#pragma unroll
    for (int i = 0; i < 6; ++i) cw[i] |= (cw[7] << (6 - i)) & 0x40404040u;
    cw[6] |= (cw[7] >> 1) & 0x40404040u;
    unsigned total = 0;
    for (;;) {
      const unsigned long long bal = __ballot(M != 0);
      if (bal == 0 || total > 128) break;
      if (M != 0) {
        const unsigned jb = __builtin_ctz(M); M &= M - 1;
        const unsigned pos = (jb >> 4) * 1024 + lane * 16 + (jb & 15);
        const unsigned e = (scr[pos] >> 3) & 15u, x = e ^ (((e - base) & 7u) + base);
        const unsigned idx = total + __builtin_amdgcn_mbcnt_hi((unsigned)(bal >> 32), __builtin_amdgcn_mbcnt_lo((unsigned)bal, 0u));
        if (idx < 128) *(LAS unsigned short*)(scr + 2048 + idx * 2) = (unsigned short)(((wid & 1) << 11 | pos) << 4 | x);
      }
      total += (unsigned)__builtin_popcountll(bal);
    }
    if (lane == 0) ctl[8 + wid] = total;
    const unsigned ent0 = *(LAS unsigned short*)(scr + 2048 + lane * 2), ent1 = *(LAS unsigned short*)(scr + 2048 + 128 + lane * 2);
    asm volatile("s_waitcnt lgkmcnt(0)\n\ts_barrier" ::: "memory");
    const unsigned t0_ = ctl[8 + (wid & 6)], t1_ = ctl[9 + (wid & 6)];
    const unsigned off = (wid & 1) ? t0_ : 0;
    const bool raw = t0_ + t1_ > 128;
    const unsigned U = ((p * 2 + op) * 64 + G) * 4 + (wid & 3);
    if (!raw) {
      char* pk = sb + SRC_PACK + (size_t)U * 1792;
      st16_sys(pk + lane * 16, (u32x4){cw[0], cw[1], cw[2], cw[3]});
      { typedef unsigned u32x3 __attribute__((ext_vector_type(3))); const u32x3 v3 = {cw[4], cw[5], cw[6]};
        asm volatile("global_store_dwordx3 %0, %1, off sc0 sc1\n\ts_nop 1" :: "v"(pk + 1024 + lane * 12), "v"(v3) : "memory"); }
      char* ex = sb + SRC_EXC + (size_t)(U >> 1) * 256 + (off + lane) * 2;
      if (lane < (int)total) asm volatile("global_store_short %0, %1, off sc0 sc1\n\ts_nop 1" :: "v"(ex), "v"(ent0) : "memory");
      if (lane + 64 < (int)total) asm volatile("global_store_short %0, %1, off offset:128 sc0 sc1\n\ts_nop 1" :: "v"(ex), "v"(ent1) : "memory");
    }
    if (lane == 0) asm volatile("global_store_short %0, %1, off sc0 sc1\n\ts_nop 1" :: "v"(sb + SRC_HDR + U * 2), "v"(raw ? 0x8000u : ((t0_ + t1_) | base << 8)) : "memory");
    asm volatile("s_waitcnt vmcnt(0)\n\ts_barrier" ::: "memory");
    if (tid < NR) st4_sys(flagp + (WS_READY2 - WS_READY), call_);
  }
  const unsigned call = (unsigned)__builtin_amdgcn_readfirstlane(ctl[4]);

  if (b < GEMM0) {
    __syncthreads();
    if (tid < 32) ctl[16 + tid] = 0;
    asm volatile("s_waitcnt lgkmcnt(0)\n\ts_barrier" ::: "memory");
    const unsigned lctl = (unsigned)(size_t)lds + CTL_OFF;
    if (b >= NPULL) {
      if (b != NPULL || wid != 7) return;
      const unsigned pbase = call * 256;
      const int mp = lane >> 3, mo = lane & 7;
      unsigned* pl = (unsigned*)(myws + WS_PROG + mo * 256) + mp;
      const unsigned long long t0 = now();
      {
        const unsigned* fl = (const unsigned*)(myws + WS_READY) + rank * 256 + lane * 4;
        for (;;) {
          u32x4 v; asm volatile("global_load_dwordx4 %0, %1, off sc0 sc1\n\ts_waitcnt vmcnt(0)" : "=&v"(v) : "v"(fl) : "memory");
          if (__all(v[0] >= call && v[1] >= call && v[2] >= call && v[3] >= call)) break;
          if (give_up(t0, tmo, call)) break;
          __builtin_amdgcn_s_sleep(2);
        }
        if (mp == 7) st4_sys(pl, pbase + 128);
      }
      const unsigned* cnt = (const unsigned*)(myws + WS_CNT) + (mp < 7 ? mp : 0) * 128;
      int pos = mp < 7 ? 0 : 128;
      for (;;) {
        const int q = pos + mo < 128 ? pos + mo : 127;
        unsigned v; asm volatile("global_load_dword %0, %1, off sc1\n\ts_waitcnt vmcnt(0)" : "=&v"(v) : "v"(cnt + q) : "memory");
        const unsigned long long m = __ballot(mp < 7 && pos + mo < 128 && v >= call);
        const int adv = __builtin_ctz(~(unsigned)((m >> (mp * 8)) & 0xFF));
        if (adv > 0) { pos += adv; st4_sys(pl, pbase + pos); }
        if (__all(pos >= 128)) break;
        if (!__any(adv > 0)) { if (give_up(t0, tmo, call)) break; __builtin_amdgcn_s_sleep(2); }
      }
      return;
    }
    const int pi = b % 7, peer = (rank + 1 + pi) & 7;
    auto lds_rd = [&](unsigned a) -> unsigned { unsigned v; asm volatile("ds_read_b32 %0, %1\n\ts_waitcnt lgkmcnt(0)" : "=v"(v) : "v"(a) : "memory"); return (unsigned)__builtin_amdgcn_readfirstlane(v); };
    const char* srcb = P.ws_all[peer] + WS_SRC + (size_t)(call & 1) * SRC_BYTES;
    const unsigned lhdr = (unsigned)(size_t)lds + 16384;
    if (wid == 7) {
      unsigned* tk = (unsigned*)(myws + WS_TKT + pi * 128);
      const unsigned base = (call - 1) * (132 - 4 * NSTATIC) - 4 * NSTATIC;
      const unsigned long long t0 = now();
      const unsigned* rdy = (const unsigned*)(myws + WS_READY) + peer * 256 + lane * 4;
      const unsigned* rdy2 = (const unsigned*)(myws + WS_READY2) + peer * 256 + lane * 4;
      unsigned long long rmask = 0;
      unsigned hdr_ok = 0;
      for (int t = 0;; ++t) {
        if (t >= AHEAD) while (lds_rd(lctl + 4 * (16 + ((t - AHEAD) & 7))) != 8 && now() - t0 < 2 * DEADLINE) __builtin_amdgcn_s_sleep(1);
        unsigned v = (unsigned)(b / 7 + 4 * t);
        if (t >= NSTATIC) {
          if (lane == 0) asm volatile("global_atomic_add %0, %1, %2, off sc0 sc1\n\ts_waitcnt vmcnt(0)" : "=&v"(v) : "v"(tk), "v"(1u) : "memory");
          v = (unsigned)__builtin_amdgcn_readfirstlane(v) - base;
        }
        const bool stop = v >= 128 || t >= 254;
        if (!stop && rmask != ~0ull) {
          const unsigned o = chunk_off((int)v, pi);
          const int bit = (o >> 20) * 16 + ((o >> 15) & 15);
          if (!((rmask >> bit) & 1)) {
            const unsigned long long t1 = now();
            for (;;) {
              u32x4 f; asm volatile("global_load_dwordx4 %0, %1, off sc0 sc1\n\ts_waitcnt vmcnt(0)" : "=&v"(f) : "v"(rdy) : "memory");
              rmask = __ballot(f[0] >= call && f[1] >= call && f[2] >= call && f[3] >= call);
              if ((rmask >> bit) & 1) break;
              if (give_up(t1, tmo, call)) { rmask = ~0ull; break; }
              __builtin_amdgcn_s_sleep(2);
            }
          }
        }
        const bool packed = !stop && ((hdr_ok >> (chunk_off((int)v, pi) >> 20)) & 1);
        if (lane == 0) asm volatile("ds_write_b32 %0, %1\n\tds_write_b32 %2, %3" :: "v"(lctl + 4 * (16 + (t & 7))), "v"(0u), "v"(lctl + 4 * (32 + (t & 7))),
                                    "v"((unsigned)(t + 1) << 16 | (packed ? 0u : 256u) | (stop ? 255u : v + 1)) : "memory");
        if (stop) break;
        if (hdr_ok != 15 && t >= 2) {
          u32x4 f; asm volatile("global_load_dwordx4 %0, %1, off sc0 sc1\n\ts_waitcnt vmcnt(0)" : "=&v"(f) : "v"(rdy2) : "memory");
          const unsigned long long m2 = __ballot(f[0] >= call && f[1] >= call && f[2] >= call && f[3] >= call);
          unsigned newp = 0;
#pragma unroll
          for (int pp = 0; pp < 4; ++pp) if (((m2 >> (16 * pp)) & 0xFFFF) == 0xFFFF && !((hdr_ok >> pp) & 1)) newp |= 1u << pp;
          if (newp) {
            u32x4 h0, h1, h2, h3; const char* hp = srcb + SRC_HDR + lane * 16;
            asm volatile("global_load_dwordx4 %0, %4, off sc0 sc1\n\tglobal_load_dwordx4 %1, %4, off offset:1024 sc0 sc1\n\t"
                         "global_load_dwordx4 %2, %4, off offset:2048 sc0 sc1\n\tglobal_load_dwordx4 %3, %4, off offset:3072 sc0 sc1\n\ts_waitcnt vmcnt(0)"
                         : "=&v"(h0), "=&v"(h1), "=&v"(h2), "=&v"(h3) : "v"(hp) : "memory");
            if (newp & 1) asm volatile("ds_write_b128 %0, %1" :: "v"(lhdr + lane * 16), "v"(h0) : "memory");
            if (newp & 2) asm volatile("ds_write_b128 %0, %1 offset:1024" :: "v"(lhdr + lane * 16), "v"(h1) : "memory");
            if (newp & 4) asm volatile("ds_write_b128 %0, %1 offset:2048" :: "v"(lhdr + lane * 16), "v"(h2) : "memory");
            if (newp & 8) asm volatile("ds_write_b128 %0, %1 offset:3072" :: "v"(lhdr + lane * 16), "v"(h3) : "memory");
            asm volatile("s_waitcnt lgkmcnt(0)" ::: "memory");
            hdr_ok |= newp;
          }
        }
      }
      return;
    }
    char* dstb = myws + WS_DST + (size_t)peer * SHARD_BYTES;
    unsigned* cntb = (unsigned*)(myws + WS_CNT) + pi * 128;
    const unsigned scr = (unsigned)(size_t)lds + wid * 2048;
    struct Pair { int t, q; unsigned hdr, off; };
    auto fetch = [&](int m, Pair& f) -> bool {
      const int t = m >> 3;
      unsigned v = lds_rd(lctl + 4 * (32 + (t & 7)));
      if ((v >> 16) != (unsigned)(t + 1)) {
        const unsigned long long t0 = now();
        while (((v = lds_rd(lctl + 4 * (32 + (t & 7)))) >> 16) != (unsigned)(t + 1)) { if (now() - t0 > 2 * DEADLINE) return false; __builtin_amdgcn_s_sleep(1); }
      }
      if ((v & 255) == 255) return false;
      f.t = t; f.q = (int)(v & 255) - 1;
      const unsigned o = chunk_off(f.q, pi), i = m & 7;
      f.off = o + (i >> 1) * 8192 + (i & 1) * 4096;
      f.hdr = (v & 256) ? 0x80008000u : lds_rd(lhdr + ((o >> 11) + 2 * i) * 2);
      return true;
    };
    const char* locb = myws + WS_SRC + (size_t)(call & 1) * SRC_BYTES;
#define LD2(RA, RB, RB3) "global_load_dwordx4 " RA ", %[l16], %[pa] sc0 sc1\n\t" \
                   "s_cmp_lg_u32 %[raw], 0\n\ts_cbranch_scc1 1f\n\t" \
                   "global_load_dwordx3 " RB3 ", %[l12], %[pa] offset:1024 sc0 sc1\n\ts_branch 2f\n" \
                   "1:\n\tglobal_load_dwordx4 " RB ", %[l16], %[pa] offset:2048 sc0 sc1\n" \
                   "2:"
#define UNIT_INDEX(off_, u_) ((((off_) >> 15) << 4) + ((((off_) >> 13) & 3) << 2) + ((((off_) >> 12) & 1) << 1) + (u_))
#define ISSUE0(f_, valid_) do {     \
      const unsigned h_ = UNI((f_).hdr), o_ = UNI((f_).off), U_ = UNIT_INDEX(o_, 0), raw_ = h_ & 0x8000u, cnt_ = raw_ ? 0u : h_ & 255; \
      const char* sb_ = (valid_) ? srcb : locb; \
      const char* pa_ = sb_ + (raw_ ? SRC_RAW + o_ : SRC_PACK + (size_t)U_ * 1792); \
      const char* ep_ = sb_ + SRC_EXC + (size_t)(U_ >> 1) * 256; const unsigned eo_ = min((unsigned)lane, cnt_ > 1 ? (cnt_ - 1) >> 1 : 0u) * 4; \
      asm volatile(LD2("v[200:203]", "v[204:207]", "v[204:206]") "\n\tglobal_load_dword v208, %[eo], %[pe] sc0 sc1" \
                   : "={v[200:203]}"(a0), "={v[204:207]}"(b0), "={v208}"(ee) \
                   : [l16] "v"(lane * 16), [l12] "v"(lane * 12), [eo] "v"(eo_), [pa] "s"(pa_), [pe] "s"(ep_), [raw] "s"(raw_) : "scc", "memory"); \
    } while (0)
#define ISSUE1(f_, valid_) do {     \
      const unsigned h_ = UNI((f_).hdr) >> 16, o_ = UNI((f_).off), U_ = UNIT_INDEX(o_, 1), raw_ = h_ & 0x8000u; \
      const char* sb_ = (valid_) ? srcb : locb; \
      const char* pa_ = sb_ + (raw_ ? SRC_RAW + o_ + 1024 : SRC_PACK + (size_t)U_ * 1792); \
      asm volatile(LD2("v[210:213]", "v[214:217]", "v[214:216]") \
                   : "={v[210:213]}"(a1), "={v[214:217]}"(b1) \
                   : [l16] "v"(lane * 16), [l12] "v"(lane * 12), [pa] "s"(pa_), [raw] "s"(raw_) : "scc", "memory"); \
    } while (0)
    auto arrive = [&](const Pair& f) {
      unsigned old = 0;
      if (lane == 0) asm volatile("ds_add_rtn_u32 %0, %1, %2\n\ts_waitcnt lgkmcnt(0)" : "=&v"(old) : "v"(lctl + 4 * (16 + (f.t & 7))), "v"(1u) : "memory");
      if (__builtin_amdgcn_readfirstlane(old) == 7 && lane == 0) st4_sys(cntb + f.q, call);
    };
    auto process = [&](u32x4 a, u32x4 bq, unsigned e, const Pair& f, int u) {
      u32x4 oa = a, ob = bq;
      const unsigned h0 = UNI(f.hdr), fh = u ? h0 >> 16 : h0;
      if (!(fh & 0x8000u)) {
        const unsigned base8 = ((fh >> 8) & 15) * 0x08080808u, cnt = h0 & 255;
        unsigned c7 = (bq[2] << 1) & 0x80808080u;
        c7 |= (a[0] >> 6) & 0x01010101u; c7 |= (a[1] >> 5) & 0x02020202u; c7 |= (a[2] >> 4) & 0x04040404u; c7 |= (a[3] >> 3) & 0x08080808u;
        c7 |= (bq[0] >> 2) & 0x10101010u; c7 |= (bq[1] >> 1) & 0x20202020u;
#pragma unroll
        for (int i = 0; i < 4; ++i) oa[i] = (a[i] & 0xBFBFBFBFu) + base8;
#pragma unroll
        for (int i = 0; i < 3; ++i) ob[i] = (bq[i] & 0xBFBFBFBFu) + base8;
        ob[3] = c7 + base8;
        if (cnt > 0) {
          const unsigned e0 = e & 0xFFFFu, e1 = e >> 16;
          asm volatile("ds_write_b128 %2, %0\n\tds_write_b128 %2, %1 offset:1024" :: "v"(oa), "v"(ob), "v"(scr + lane * 16) : "memory");
          if (2u * lane < cnt && (int)(e0 >> 15) == u) asm volatile("ds_xor_b32 %0, %1" :: "v"(scr + ((e0 >> 4) & 0x7FCu)), "v"((e0 & 15u) << (3 + 8 * ((e0 >> 4) & 3))) : "memory");
          if (2u * lane + 1 < cnt && (int)(e1 >> 15) == u) asm volatile("ds_xor_b32 %0, %1" :: "v"(scr + ((e1 >> 4) & 0x7FCu)), "v"((e1 & 15u) << (3 + 8 * ((e1 >> 4) & 3))) : "memory");
          asm volatile("ds_read_b128 %0, %2\n\tds_read_b128 %1, %2 offset:1024\n\ts_waitcnt lgkmcnt(0)" : "=&v"(oa), "=&v"(ob) : "v"(scr + lane * 16) : "memory");
        }
      }
      char* dp = dstb + UNI(f.off) + u * 1024;
      asm volatile("global_store_dwordx4 %0, %1, %3 sc0 sc1\n\tglobal_store_dwordx4 %0, %2, %3 offset:2048 sc0 sc1\n\ts_nop 1" :: "v"(lane * 16), "v"(oa), "v"(ob), "s"(dp) : "memory");
    };
    if (wid == 0) asm volatile("global_load_dword v250, %0, %1 sc0 sc1\n\tglobal_load_dword v250, %0, %2 sc0 sc1" :: "v"(lane * 65536u), "s"(srcb), "s"(srcb + SHARD_BYTES) : "memory", "v250");
    if (wid == 1) asm volatile("global_load_dword v250, %0, %1 sc0 sc1" :: "v"(lane * 65536u), "s"(dstb) : "memory", "v250");
    u32x4 a0, b0, a1, b1; unsigned ee;
    Pair f = {0, 0, 0x80008000u, 0u}, fn = f, prev = f;
    bool v = fetch(wid, f), have_prev = false;
    if (v) {
      ISSUE0(f, true); ISSUE1(f, true);
      asm volatile("s_waitcnt vmcnt(2)" : "+{v[200:203]}"(a0), "+{v[204:207]}"(b0), "+{v208}"(ee) :: "memory");
      for (int m = wid;; m += 7) {
        const unsigned e = ee;
        unsigned ecur = e; asm volatile("" : "+v"(ecur));
        process(a0, b0, ecur, f, 0);
        const bool vn = fetch(m + 7, fn);
        ISSUE0(fn, vn);
        asm volatile("s_waitcnt vmcnt(5)" : "+{v[210:213]}"(a1), "+{v[214:217]}"(b1) :: "memory");
        if (have_prev) arrive(prev);
        process(a1, b1, ecur, f, 1);
        ISSUE1(fn, vn);
        asm volatile("s_waitcnt vmcnt(4)" : "+{v[200:203]}"(a0), "+{v[204:207]}"(b0), "+{v208}"(ee) :: "memory");
        prev = f; have_prev = true;
        if (!vn) break;
        f = fn;
      }
      asm volatile("s_waitcnt vmcnt(0)" ::: "memory");
      arrive(prev);
    }
#undef ISSUE0
#undef ISSUE1
#undef LD2
#undef UNIT_INDEX
    return;
  }

  const float scale = P.sx[0] * P.sw[0];
  const float inv_scale = 1.0f / scale;
  const int g = b - GEMM0, xcd = g & 7, idx = g >> 3;
  const int pm = (xcd & 1) * 7 + idx % 7, pn = (xcd >> 1) * 4 + idx / 7;
  const int g0 = tile_start(pm), g1 = tile_start(pm + 1), wr = wid >> 2, wc = wid & 3;
  const bool a4 = wid < (g1 - g0) * 2 - 32;
  const int split = g1 - g0 == 19 ? 10 : 9;
  const int skip9 = wr + (19 - (g1 - g0));
  const char* own = myws + WS_SRC + (size_t)(call & 1) * SRC_BYTES + SRC_RAW;
  const char* dstb = myws + WS_DST;
  const size_t aoff = (size_t)g0 * 2048, boff = OP_BYTES + (size_t)pn * B_BYTES;
  auto kt_base = [&](int kt) -> const char* { const int s = kt >> 2, p = kt & 3; return (s == rank ? own : dstb + (size_t)s * SHARD_BYTES) + (size_t)p * KT_BYTES; };

  unsigned pollA, thr;
  {
    const int kt = lane & 31, s = kt >> 2, p = kt & 3, pi = (s - rank - 1) & 7;
    const int qa = chunk_pos(p, 0, g0 >> 4, pi), qb = chunk_pos(p, 0, (g1 - 1) >> 4, pi), qc = chunk_pos(p, 1, pn, pi);
    pollA = (unsigned)WS_PROG + (unsigned)(g & (NREP - 1)) * 256 + pi * 4;
    thr = call * 256 + (pi == 7 ? 128 : max(max(qa, qb), qc) + 1);
  }

  f32x4 acc[10][4];
#pragma unroll
  for (int m = 0; m < 10; ++m)
#pragma unroll
    for (int n = 0; n < 4; ++n) acc[m][n] = (f32x4){0.f, 0.f, 0.f, 0.f};

  const unsigned lbase = (unsigned)(size_t)lds;
  const unsigned voff = wid * 1024 + lane * 16;
  unsigned pa = lbase + wr * split * 2048 + lane * 16, pb = lbase + A_BYTES + wc * (4 * 2048) + lane * 16;
  unsigned npa = pa + BUF_BYTES, npb = pb + BUF_BYTES;
  const unsigned ctl_addr = lbase + CTL_OFF;
  unsigned ready = 0, taken = 0;
  auto wait_tile = [&]() {
    if (wid == 0) {
      const unsigned long long t0 = now();
      for (;;) {
        unsigned q; asm volatile("global_load_dword %0, %1, %2 sc1\n\ts_waitcnt vmcnt(0)" : "=&v"(q) : "v"(pollA), "s"(myws) : "memory");
        unsigned m = (unsigned)__ballot(q >= thr);
        if ((m & ~taken) == 0 && give_up(t0, tmo, call)) m = ~0u;
        if ((m & ~taken) != 0) { if (lane == 0) asm volatile("ds_write_b32 %0, %1" :: "v"(ctl_addr + 8), "v"(m) : "memory"); break; }
        __builtin_amdgcn_s_sleep(4);
      }
    }
    unsigned mr;
    asm volatile("s_waitcnt lgkmcnt(0)\n\ts_barrier\n\tds_read_b32 %0, %1\n\ts_waitcnt lgkmcnt(0)" : "=v"(mr) : "v"(ctl_addr + 8) : "memory");
    ready = (unsigned)__builtin_amdgcn_readfirstlane(mr);
  };
  if (wid == 1) {
    const char* ob = (const char*)P.out + (size_t)g0 * 16 * (NDIM * 4) + pn * 1024;
#pragma unroll
    for (int i = 0; i < 5; ++i) { const unsigned row = min(i * 64 + lane, (g1 - g0) * 16 - 1); asm volatile("global_load_dword v255, %0, %1" :: "v"(row * (NDIM * 4u)), "s"(ob) : "memory", "v255"); }
  }
  wait_tile();
  Regs r;
  { const int k = __builtin_ctz(ready); taken |= 1u << k; const char* kb = kt_base(k);
    stage_all(kb + aoff, kb + boff, voff, lbase + wid * 1024, a4);
    asm volatile("s_waitcnt vmcnt(0)\n\ts_barrier" ::: "memory");
    prime_reads(r, pa, pb); }
  int cur = 0;
  for (int n = 0; n < 31; ++n) {
    const unsigned nl = lbase + (cur ^ 1) * BUF_BYTES + wid * 1024;
    const unsigned la = nl, lb = nl + A_BYTES;
    unsigned q0;
    const unsigned ctl_cur = ctl_addr + (n & 1) * 4;
#define DB(i) dma(gb + (i) * 8192, voff, lb + (i) * 8192)
#define DA(i) dma(ga + (i) * 8192, voff, la + (i) * 8192)
    auto try_stage = [&]() -> bool {
      const unsigned cand = ready & ~taken;
      if (cand == 0) return false;
      const int k = __builtin_ctz(cand); taken |= 1u << k;
      if (wr == 1) {
        const char* kb = kt_base(k);
        const char* ga = kb + aoff; const char* gb = kb + boff;
        const unsigned vo2 = voff - 4096, la2 = la - 4096, lb2 = lb - 4096;
        dma(gb, vo2, lb2); DB(0); dma(gb + 8192, vo2, lb2 + 8192); DB(1); dma(gb + 2 * 8192, vo2, lb2 + 2 * 8192); DB(2); dma(gb + 3 * 8192, vo2, lb2 + 3 * 8192); DB(3);
        dma(ga, vo2, la2); DA(0); dma(ga + 8192, vo2, la2 + 8192); DA(1); dma(ga + 2 * 8192, vo2, la2 + 2 * 8192); DA(2); dma(ga + 3 * 8192, vo2, la2 + 3 * 8192); DA(3);
        dma(ga + 4 * 8192, vo2, la2 + 4 * 8192); if (a4) DA(4);
      }
      return true;
    };
#define POLL_ISSUE() do { if (wid == 0) asm volatile("global_load_dword %0, %1, %2 sc1" : "=&v"(q0) : "v"(pollA), "s"(myws) : "memory"); } while (0)
#define POLL_PUBLISH(addr) do { if (wid == 0) {   \
      asm volatile("s_waitcnt vmcnt(0)" : "+v"(q0) :: "memory"); \
      const unsigned m_ = (unsigned)__ballot(q0 >= thr); \
      if (lane == 0) asm volatile("ds_write_b32 %0, %1" :: "v"(addr), "v"(m_) : "memory"); } } while (0)
    bool stage = try_stage();
    const bool nopoll = (ready & ~taken) != 0 || taken == ~0u;
    if (!nopoll) POLL_ISSUE();
#define CHECKPOINT(word) do { if (!stage) { \
      POLL_PUBLISH(ctl_addr + (word)); \
      unsigned mm; \
      asm volatile("s_waitcnt lgkmcnt(0)\n\ts_barrier\n\tds_read_b32 %0, %1\n\ts_waitcnt lgkmcnt(0)" : "=v"(mm) : "v"(ctl_addr + (word)) : "memory"); \
      ready = (unsigned)__builtin_amdgcn_readfirstlane(mm); \
      POLL_ISSUE(); \
      stage = try_stage(); } } while (0)
    __builtin_amdgcn_s_setprio(1);
    WL(10); MM(acc[0][0], R_B0, r.b0, R_AX, r.ax);
    WL(8); MM(acc[0][1], R_B1, r.b1, R_AX, r.ax);
    WL(6); MM(acc[0][2], R_B2, r.b2, R_AX, r.ax);
    WL(4); MM(acc[0][3], R_B3, r.b3, R_AX, r.ax);
    RD_AX(pa, 3 * 2048);
    WL(4); MMA4Y(1); RD_AY(pa, 4 * 2048);
    WL(4); MMA4Z(2); RD_AZ(pa, 5 * 2048);
    CHECKPOINT(12);
    WL(4); MMA4X(3); RD_AX(pa, 6 * 2048);
    WL(4); MMA4Y(4); RD_AY(pa, 7 * 2048);
    WL(4); MMA4Z(5); RD_AZ(pa, 8 * 2048);
    if (stage && !nopoll) POLL_ISSUE();
    CHECKPOINT(16);
    WL(4); MMA4X(6);
    WL(2); MMA4Y(7); RD_AY(pa, 9 * 2048);
    WL(2); MMA4Z(8);
    if (!nopoll) POLL_PUBLISH(ctl_cur);
#undef CHECKPOINT
    asm volatile("s_waitcnt vmcnt(0) lgkmcnt(0)\n\ts_barrier" ::: "memory");
    unsigned mreg;
    asm volatile("ds_read_b32 %0, %1" : "=v"(mreg) : "v"(ctl_cur));
    RD_AX(npa, 0);
    asm volatile(
      "s_cmp_lg_u32 %[wr], 0\n\t"
      "s_cbranch_scc1 1f\n\t"
      "v_mfma_f32_16x16x128_f8f6f4 %[c0], v[240:247], v[200:207], %[c0]\n\t"
      "ds_read_b128 v[200:203], %[pb] offset:0\n\tds_read_b128 v[204:207], %[pb] offset:1024\n\t"
      "v_mfma_f32_16x16x128_f8f6f4 %[c1], v[240:247], v[208:215], %[c1]\n\t"
      "ds_read_b128 v[208:211], %[pb] offset:2048\n\tds_read_b128 v[212:215], %[pb] offset:3072\n\t"
      "v_mfma_f32_16x16x128_f8f6f4 %[c2], v[240:247], v[216:223], %[c2]\n\t"
      "ds_read_b128 v[216:219], %[pb] offset:4096\n\tds_read_b128 v[220:223], %[pb] offset:5120\n\t"
      "v_mfma_f32_16x16x128_f8f6f4 %[c3], v[240:247], v[224:231], %[c3]\n\t"
      "ds_read_b128 v[224:227], %[pb] offset:6144\n\tds_read_b128 v[228:231], %[pb] offset:7168\n\t"
      "s_branch 2f\n"
      "1:\n\t"
      "ds_read_b128 v[200:203], %[pb] offset:0\n\tds_read_b128 v[204:207], %[pb] offset:1024\n\t"
      "ds_read_b128 v[208:211], %[pb] offset:2048\n\tds_read_b128 v[212:215], %[pb] offset:3072\n\t"
      "ds_read_b128 v[216:219], %[pb] offset:4096\n\tds_read_b128 v[220:223], %[pb] offset:5120\n\t"
      "ds_read_b128 v[224:227], %[pb] offset:6144\n\tds_read_b128 v[228:231], %[pb] offset:7168\n"
      "2:"
      : [c0] "+v"(acc[9][0]), [c1] "+v"(acc[9][1]), [c2] "+v"(acc[9][2]), [c3] "+v"(acc[9][3]),
        "+{v[200:207]}"(r.b0), "+{v[208:215]}"(r.b1), "+{v[216:223]}"(r.b2), "+{v[224:231]}"(r.b3)
      : [wr] "s"(skip9), [pb] "v"(npb), "{v[240:247]}"(r.ay) : "scc");
    RD_AY(npa, 2048); RD_AZ(npa, 2 * 2048);
    __builtin_amdgcn_s_setprio(0);
    asm volatile("s_waitcnt lgkmcnt(14)" : "+v"(mreg));
    if (!nopoll) ready = (unsigned)__builtin_amdgcn_readfirstlane(mreg);
#undef DA
#undef DB
#undef POLL_ISSUE
#undef POLL_PUBLISH
    if (!stage) {
      if ((ready & ~taken) == 0) wait_tile();
      const int k2 = __builtin_ctz(ready & ~taken); taken |= 1u << k2;
      const char* kb2 = kt_base(k2);
      stage_all(kb2 + aoff, kb2 + boff, voff, nl, a4);
      asm volatile("s_waitcnt vmcnt(0)\n\ts_barrier" ::: "memory");
      prime_reads(r, npa, npb);
    }
    { unsigned t = pa; pa = npa; npa = t; t = pb; pb = npb; npb = t; cur ^= 1; }
  }
  {
    typedef float f32x2 __attribute__((ext_vector_type(2)));
    unsigned tq;
    asm volatile("global_load_dword %0, %1, off sc0 sc1" : "=&v"(tq) : "v"(tmo) : "memory");
    const float kk = scale * -1.44269504088896340736f, lim = 60.0f * 1.44269504088896340736f;
    const f32x2 kk2 = {kk, kk};
    f32x2 is2;
    const int fr = lane & 15, fq = lane >> 4;
    const unsigned ovoff = fq * (4 * NDIM * 4) + fr * 16;
    char* obase = (char*)P.out + (size_t)(g0 + wr * split) * 16 * (NDIM * 4) + pn * 1024 + wc * 256;
    auto group = [&](const f32x4 (&a4)[4], int m) {
      SB();
      f32x4 o[4];
#pragma unroll
      for (int n = 0; n < 4; ++n)
#pragma unroll
        for (int jp = 0; jp < 4; jp += 2) {
          const f32x2 a = {a4[n][jp], a4[n][jp + 1]};
          const f32x2 t = a * kk2;
          f32x2 e = {__builtin_amdgcn_exp2f(__builtin_amdgcn_fmed3f(t[0], -lim, lim)), __builtin_amdgcn_exp2f(__builtin_amdgcn_fmed3f(t[1], -lim, lim))};
          const f32x2 d = __builtin_elementwise_fma(e, is2, is2);
          o[jp][n] = a[0] * __builtin_amdgcn_rcpf(d[0]); o[jp + 1][n] = a[1] * __builtin_amdgcn_rcpf(d[1]);
        }
      char* ob = obase + (size_t)m * 16 * (NDIM * 4);
#define OUT_ST(POL) asm volatile("global_store_dwordx4 %0, %1, %5" POL "\n\tglobal_store_dwordx4 %0, %2, %6" POL "\n\tglobal_store_dwordx4 %0, %3, %7" POL "\n\tglobal_store_dwordx4 %0, %4, %8" POL "\n\ts_nop 1" \
                   :: "v"(ovoff), "v"(o[0]), "v"(o[1]), "v"(o[2]), "v"(o[3]), "s"(ob), "s"(ob + NDIM * 4), "s"(ob + 2 * NDIM * 4), "s"(ob + 3 * NDIM * 4) : "memory")
      OUT_ST(" sc1");
#undef OUT_ST
      SB();
    };
    WL(10); MM(acc[0][0], R_B0, r.b0, R_AX, r.ax);
    WL(8); MM(acc[0][1], R_B1, r.b1, R_AX, r.ax);
    WL(6); MM(acc[0][2], R_B2, r.b2, R_AX, r.ax);
    WL(4); MM(acc[0][3], R_B3, r.b3, R_AX, r.ax);
    RD_AX(pa, 3 * 2048);
    WL(4); MMA4Y(1); RD_AY(pa, 4 * 2048);
    WL(4); MMA4Z(2); RD_AZ(pa, 5 * 2048);
    asm volatile("s_waitcnt vmcnt(0)" : "+v"(tq) :: "memory");
    { const float is = (unsigned)__builtin_amdgcn_readfirstlane(tq) == call ? __builtin_nanf("") : inv_scale; is2 = (f32x2){is, is}; }
    group(acc[0], 0);
    WL(4); MMA4X(3); RD_AX(pa, 6 * 2048);
    group(acc[1], 1);
    WL(4); MMA4Y(4); RD_AY(pa, 7 * 2048);
    group(acc[2], 2);
    WL(4); MMA4Z(5); RD_AZ(pa, 8 * 2048);
    group(acc[3], 3);
    WL(4); MMA4X(6);
    group(acc[4], 4);
    WL(2); MMA4Y(7); RD_AY(pa, 9 * 2048);
    group(acc[5], 5);
    WL(2); MMA4Z(8);
    group(acc[6], 6);
    if (skip9 == 0) { WL(0); MMA4Y(9); }
    group(acc[7], 7);
    SB(); asm volatile("s_nop 15\n\ts_nop 15" ::: "memory");
    group(acc[8], 8);
    if (skip9 == 0) group(acc[9], 9);
  }
}

}

extern "C" void kernel_launch(int rank, int nranks, ncclComm_t comm, hipStream_t stream,
                              void* const* d_in, const int* in_sizes, int n_in,
                              void* d_out, int out_size, const int* in_dtypes, int out_dtype,
                              void* const* d_out_all, void* const* d_ws_all, size_t ws_bytes) {
  (void)comm; (void)in_sizes; (void)n_in; (void)out_size; (void)in_dtypes; (void)out_dtype; (void)d_out_all;
  if (nranks != NR || (unsigned)rank >= (unsigned)NR || ws_bytes < WS_END) { fprintf(stderr, "unsupported configuration\n"); return; }
  static bool attr_done[NR] = {};
  if (!attr_done[rank]) {
    HIP_CHECK(hipFuncSetAttribute((const void*)fused_kernel, hipFuncAttributeMaxDynamicSharedMemorySize, LDS_BYTES));
    attr_done[rank] = true;
  }
  Params p;
  p.x = (const unsigned char*)d_in[0]; p.w = (const unsigned char*)d_in[1]; p.sx = (const float*)d_in[2]; p.sw = (const float*)d_in[3];
  p.out = (float*)d_out; p.ws_all = (char* const*)d_ws_all; p.rank = rank; p.pad = 0;
  hipLaunchKernelGGL(fused_kernel, dim3(NWG), dim3(512), LDS_BYTES, stream, p);
  HIP_CHECK(hipGetLastError());
}
```
